```python
import numpy as np

kernel_name = "gpu_ag_gemm_m4096_k4096_n16384_e5m2e5m2_dqsilu"


ABSMAX_RTOL = 0.05

COMPUTE_DTYPE = "float32"
EPILOGUE = "dequant_silu"
IN_QUANT = None
OUT_QUANT = None
IN_FP8 = "e5m2e5m2"
IN_DTYPES = {"x": "e5m2", "w_mat": "e5m2",
             "scale_x": "float32", "scale_w": "float32"}
WORLD_SIZE = 8

def _snap_fp8(x, mant_bits, max_abs, min_normal):
    a = np.abs(x.astype(np.float64))
    _m, _e = np.frexp(a)
    q = np.where(a >= min_normal, np.ldexp(1.0, _e - 1 - mant_bits),
                 min_normal * 2.0 ** -mant_bits)
    snapped = np.minimum(np.round(a / q) * q, max_abs)
    return (np.sign(x) * snapped).astype(np.float32)

def setup_inputs():

    x = _snap_fp8(np.random.randn(4096, 4096).astype(np.float32), 2, 57344.0, 2.0 ** -14)
    w_mat = _snap_fp8(np.random.randn(4096, 16384).astype(np.float32), 2, 57344.0, 2.0 ** -14)
    scale_x = (0.02 * (1.0 + np.random.rand(1))).astype(np.float32)
    scale_w = (0.02 * (1.0 + np.random.rand(1))).astype(np.float32)
    return {"x": x, "w_mat": w_mat, "scale_x": scale_x, "scale_w": scale_w}

def reference(x, w_mat, scale_x, scale_w):
    acc = (x @ w_mat).astype(np.float32)
    y = (acc * (scale_x * scale_w)).astype(np.float32)
    return (y / (1.0 + np.exp(-np.clip(y, -60.0, 60.0)))).astype(np.float32)
```

```cpp
#include <hip/hip_runtime.h>
#include <rccl/rccl.h>
#include <cstdio>

#define HIP_CHECK(expr) do { hipError_t _e = (expr); if (_e != hipSuccess) { \
    fprintf(stderr, "HIP error %s at %s:%d\n", hipGetErrorString(_e), __FILE__, __LINE__); \
    return; } } while (0)

namespace {

constexpr int NR = 8;
constexpr int KDIM = 4096;
constexpr int MLOC = 512;
constexpr int NLOC = 2048;
constexpr int BM = 128, BN = 256, BK = 128;
constexpr int NSTEP = KDIM / BK;
constexpr int GRID = 256;
constexpr int NTHREADS = 512;

constexpr int A_STAGE = BM * BK;
constexpr int W_STAGE = BN * BK;
constexpr int STAGE = A_STAGE + W_STAGE;
constexpr int NSTAGE = 3;
constexpr int LDS_CTRL = NSTAGE * STAGE;
constexpr int LDS_BYTES = LDS_CTRL + 16;

constexpr size_t WS_X = 0;
constexpr size_t X_BYTES = (size_t)MLOC * KDIM;
constexpr size_t WS_FLAG = WS_X + 2 * X_BYTES;
constexpr size_t WS_CNT = WS_FLAG + NR * GRID * 4;
constexpr size_t WS_TMO = WS_CNT + GRID * 4;
constexpr size_t WS_STAMP = WS_TMO + 4096;
constexpr size_t WS_NEED = WS_STAMP + GRID * 16 * 8;

constexpr unsigned long long DEADLINE_TICKS = 100000000ull;

typedef unsigned u32x4 __attribute__((ext_vector_type(4)));
typedef int i32x2 __attribute__((ext_vector_type(2)));
typedef int i32x4 __attribute__((ext_vector_type(4)));
typedef int i32x8 __attribute__((ext_vector_type(8)));
typedef float f32x4 __attribute__((ext_vector_type(4)));
#define LAS __attribute__((address_space(3)))
#define GAS __attribute__((address_space(1)))

struct Params {
    const unsigned char* x;
    const unsigned char* w;
    const float* sx;
    const float* sw;
    float* out;
    char* ws[NR];
    int rank;
    int pad;
};

#define STAMP(i) do {} while (0)

__device__ __forceinline__ void dma2(const void* base, unsigned voff, unsigned lds) {
    unsigned keep;
    asm volatile("s_mov_b32 %0, m0\n\ts_mov_b32 m0, %3\n\ts_nop 0\n\t"
                 "global_load_lds_dwordx4 %1, %2\n\t"
                 "global_load_lds_dwordx4 %1, %2 offset:1024\n\t"
                 "s_mov_b32 m0, %0"
                 : "=&s"(keep) : "v"(voff), "s"(base), "s"(lds) : "memory");
}
__device__ __forceinline__ void dma4(const void* base, unsigned v0, unsigned v1, unsigned v2, unsigned v3, unsigned lds) {
    unsigned keep;
    asm volatile("s_mov_b32 %0, m0\n\ts_mov_b32 m0, %6\n\ts_nop 0\n\t"
                 "global_load_lds_dwordx4 %1, %5\n\t"
                 "global_load_lds_dwordx4 %2, %5 offset:1024\n\t"
                 "global_load_lds_dwordx4 %3, %5 offset:2048\n\t"
                 "global_load_lds_dwordx4 %4, %5 offset:3072\n\t"
                 "s_mov_b32 m0, %0"
                 : "=&s"(keep) : "v"(v0), "v"(v1), "v"(v2), "v"(v3), "s"(base), "s"(lds) : "memory");
}

__device__ __forceinline__ float silu_out(float acc, float s) {
    const float y = acc * s;
    const float c = __builtin_amdgcn_fmed3f(y, -60.0f, 60.0f);
    const float e = __builtin_amdgcn_exp2f(c * -1.4426950408889634f);
    return y * __builtin_amdgcn_rcpf(1.0f + e);
}

__global__ void __launch_bounds__(NTHREADS, 2) __attribute__((amdgpu_waves_per_eu(2, 2))) ag_gemm(Params p) {
    extern __shared__ __attribute__((aligned(16))) char smem_[];
    LAS char* smem = (LAS char*)smem_;
    const int b = blockIdx.x;
    const int tid = threadIdx.x;
    const int lane = tid & 63;
    const int wave = __builtin_amdgcn_readfirstlane(tid >> 6);
    char* myws = p.ws[p.rank];
    STAMP(0);

    const int grp = b & 7, idx = b >> 3;
    const unsigned src = (unsigned)(p.rank + grp) & (NR - 1);
    const int rb = idx & 3, nt = idx >> 2;
    const int n0 = nt * BN;

    const int wr0 = 16 * wave + (lane >> 4);
    const int f0 = ((lane >> 4) & 7) | ((wave & 1) << 3);
    const unsigned woff0 = (unsigned)(wr0 * NLOC + n0 + (((lane & 15) ^ f0) << 4));
    const unsigned woff1 = (unsigned)((wr0 + 4) * NLOC + n0 + (((lane & 15) ^ f0 ^ 4) << 4)) - 1024u;
    const unsigned woff2 = woff0 + 8 * NLOC - 2048u;
    const unsigned woff3 = woff1 + 8 * NLOC - 2048u;
    const unsigned lds0 = (unsigned)(size_t)smem;
    auto load_w = [&](int step, int slot) {
        dma4(p.w + (size_t)step * (BK * NLOC), woff0, woff1, woff2, woff3,
             lds0 + slot * STAGE + A_STAGE + wave * 4096);
    };

    unsigned* cntp = (unsigned*)(myws + WS_CNT) + b;
    const unsigned call = __builtin_amdgcn_readfirstlane(
        __hip_atomic_load(cntp, __ATOMIC_RELAXED, __HIP_MEMORY_SCOPE_AGENT)) + 1u;
    const size_t xoff = WS_X + (size_t)(call & 1u) * X_BYTES;
    {
        const int srb = b >> 6, j = b & 63, ks = j >> 1;
        const int piece = 2 * (j & 1) + (wave >> 2), i = wave & 3;
        const int row = srb * BM + piece * 32 + 16 * (i >> 1) + (lane & 15);
        const int col = ks * BK + 64 * (i & 1) + 16 * (lane >> 4);
        const u32x4 v = *(const u32x4*)(p.x + (size_t)row * KDIM + col);
        char* dst = myws + xoff + (size_t)(((ks * 4 + srb) * 4 + piece) * 4096 + i * 1024 + lane * 16);
        asm volatile("global_store_dwordx4 %0, %1, off sc0 sc1\n\ts_nop 1" :: "v"(dst), "v"(v) : "memory");
        asm volatile("s_waitcnt vmcnt(0)" ::: "memory");
    }
    __syncthreads();
    STAMP(1);
    if (wave == 0) {
        if (lane < NR) {
            unsigned* f = (unsigned*)(p.ws[lane] + WS_FLAG) + p.rank * GRID + b;
            __hip_atomic_store(f, call, __ATOMIC_RELAXED, __HIP_MEMORY_SCOPE_SYSTEM);
        }
        if (lane == 0) __hip_atomic_store(cntp, call, __ATOMIC_RELAXED, __HIP_MEMORY_SCOPE_AGENT);
        const unsigned* fl = (const unsigned*)(myws + WS_FLAG) + src * GRID + rb * 64 + lane;
        unsigned* tmo = (unsigned*)(myws + WS_TMO);
        const unsigned long long t0 = __builtin_amdgcn_s_memrealtime();
        while (!__all(__hip_atomic_load(fl, __ATOMIC_RELAXED, __HIP_MEMORY_SCOPE_SYSTEM) >= call)) {
            if (__builtin_amdgcn_readfirstlane(__hip_atomic_load(tmo, __ATOMIC_RELAXED, __HIP_MEMORY_SCOPE_AGENT)) == call
                || __builtin_amdgcn_s_memrealtime() - t0 > DEADLINE_TICKS) {
                if (lane == 0) __hip_atomic_store(tmo, call, __ATOMIC_RELAXED, __HIP_MEMORY_SCOPE_AGENT);
                break;
            }
            __builtin_amdgcn_s_sleep(1);
        }
        STAMP(2);
        __builtin_amdgcn_fence(__ATOMIC_ACQUIRE, "");
        asm volatile("s_waitcnt vmcnt(0)" ::: "memory");
        const unsigned bad = __hip_atomic_load(tmo, __ATOMIC_RELAXED, __HIP_MEMORY_SCOPE_AGENT) == call;
        if (lane == 0) *(LAS unsigned*)(smem + LDS_CTRL) = bad;
        asm volatile("s_waitcnt vmcnt(0) lgkmcnt(0)" ::: "memory");
        load_w(0, 0);
        load_w(1, 1);
    } else {
        load_w(0, 0);
        load_w(1, 1);
    }
    asm volatile("" ::: "memory");
    __builtin_amdgcn_s_barrier();
    asm volatile("" ::: "memory");
    STAMP(3);
    const unsigned bad = __builtin_amdgcn_readfirstlane(*(volatile LAS unsigned*)(smem + LDS_CTRL));

    const int wm = wave >> 2, wn = wave & 3;
    f32x4 acc[4][4];
#pragma unroll
    for (int i = 0; i < 4; ++i)
#pragma unroll
        for (int j = 0; j < 4; ++j) acc[i][j] = (f32x4){0.f, 0.f, 0.f, 0.f};

    if (!bad) {
        const char* asrc = p.ws[src] + xoff + (size_t)rb * A_STAGE + wave * 2048;
        const unsigned aoff = lane * 16;
        auto load_a = [&](int step, int slot) {
            dma2(asrc + (size_t)step * (4 * A_STAGE), aoff, lds0 + slot * STAGE + wave * 2048);
        };
        load_a(0, 0);
        load_a(1, 1);

        const int a_rd = wm * 8192 + lane * 16;
        const int fq = ((lane >> 1) & 7) | (((lane >> 4) & 1) << 3);
        const int w_rd = A_STAGE + (16 * (lane >> 4) + ((lane >> 1) & 7)) * 256 + 8 * (lane & 1);

        auto compute = [&](int slot) {
            LAS char* st = smem + slot * STAGE;
            i32x8 af[4], bf[4];
#pragma unroll
            for (int i = 0; i < 4; ++i) {
                const i32x4 lo = *(LAS i32x4*)(st + a_rd + i * 2048);
                const i32x4 hi = *(LAS i32x4*)(st + a_rd + i * 2048 + 1024);
                af[i] = __builtin_shufflevector(lo, hi, 0, 1, 2, 3, 4, 5, 6, 7);
            }
#pragma unroll
            for (int j = 0; j < 4; ++j) {
                LAS char* q = st + w_rd + (((4 * wn + j) ^ fq) << 4);
                const i32x2 r0 = __builtin_amdgcn_ds_read_tr8_b64_v2i32((LAS i32x2*)(q));
                const i32x2 r1 = __builtin_amdgcn_ds_read_tr8_b64_v2i32((LAS i32x2*)(q + 2048));
                const i32x2 r2 = __builtin_amdgcn_ds_read_tr8_b64_v2i32((LAS i32x2*)(q + 16384));
                const i32x2 r3 = __builtin_amdgcn_ds_read_tr8_b64_v2i32((LAS i32x2*)(q + 18432));
                bf[j] = (i32x8){r0[0], r0[1], r1[0], r1[1], r2[0], r2[1], r3[0], r3[1]};
            }
#pragma unroll
            for (int j = 0; j < 4; ++j)
#pragma unroll
                for (int i = 0; i < 4; ++i) {
                    acc[i][j] = __builtin_amdgcn_mfma_scale_f32_16x16x128_f8f6f4(af[i], bf[j], acc[i][j], 1, 1, 0, 0, 0, 0);
                }
            __builtin_amdgcn_sched_group_barrier(0x100, 14, 0);
            __builtin_amdgcn_sched_group_barrier(0x008, 1, 0);
            __builtin_amdgcn_sched_group_barrier(0x100, 4, 0);
            __builtin_amdgcn_sched_group_barrier(0x008, 1, 0);
            __builtin_amdgcn_sched_group_barrier(0x100, 4, 0);
            __builtin_amdgcn_sched_group_barrier(0x008, 1, 0);
            __builtin_amdgcn_sched_group_barrier(0x100, 2, 0);
            __builtin_amdgcn_sched_group_barrier(0x008, 13, 0);
            __builtin_amdgcn_sched_barrier(0);
        };
        auto bar = [&]() {
            asm volatile("" ::: "memory");
            __builtin_amdgcn_s_barrier();
            asm volatile("" ::: "memory");
        };

        int slot = 0;
        asm volatile("s_waitcnt vmcnt(2)" ::: "memory");
        for (int s = 0; s < NSTEP - 2; ++s) {
            bar();
            const int ns = slot == 0 ? 2 : slot - 1;
            load_a(s + 2, ns);
            load_w(s + 2, ns);
            compute(slot);
            slot = slot == 2 ? 0 : slot + 1;
            asm volatile("s_waitcnt vmcnt(6) lgkmcnt(0)" ::: "memory");
        }
        bar();
        compute((NSTEP - 2) % NSTAGE);
        asm volatile("s_waitcnt vmcnt(0)" ::: "memory");
        bar();
        STAMP(8);
        compute((NSTEP - 1) % NSTAGE);
    } else {
        asm volatile("s_waitcnt vmcnt(0)" ::: "memory");
    }
    STAMP(9);

    float s = p.sx[0] * p.sw[0];
    if (bad) s = __builtin_nanf("");
    const int m0 = (int)src * MLOC + rb * BM + wm * 64;
    float* o = p.out + (size_t)(m0 + 4 * (lane >> 4)) * NLOC + n0 + wn * 64 + (lane & 15);
#pragma unroll
    for (int i = 0; i < 4; ++i)
#pragma unroll
        for (int r = 0; r < 4; ++r)
#pragma unroll
            for (int j = 0; j < 4; ++j)
                o[(size_t)(i * 16 + r) * NLOC + j * 16] = silu_out(acc[i][j][r], s);
}

}

extern "C" void kernel_launch(int rank, int nranks, ncclComm_t comm,
                              hipStream_t stream,
                              void* const* d_in, const int* in_sizes, int n_in,
                              void* d_out, int out_size,
                              const int* in_dtypes, int out_dtype,
                              void* const* d_out_all,
                              void* const* d_ws_all, size_t ws_bytes) {
    (void)comm; (void)in_dtypes; (void)out_dtype; (void)d_out_all;
    if (nranks != NR || (unsigned)rank >= (unsigned)NR || n_in != 4 || ws_bytes < WS_NEED ||
        in_sizes[0] != MLOC * KDIM || in_sizes[1] != KDIM * NLOC || out_size != NR * MLOC * NLOC) {
        fprintf(stderr, "ag_gemm: unexpected problem shape\n");
        return;
    }
    HIP_CHECK(hipFuncSetAttribute((const void*)ag_gemm, hipFuncAttributeMaxDynamicSharedMemorySize, LDS_BYTES));
    Params p;
    p.x = (const unsigned char*)d_in[0];
    p.w = (const unsigned char*)d_in[1];
    p.sx = (const float*)d_in[2];
    p.sw = (const float*)d_in[3];
    p.out = (float*)d_out;
    for (int i = 0; i < NR; ++i) p.ws[i] = (char*)d_ws_all[i];
    p.rank = rank;
    p.pad = 0;
    hipLaunchKernelGGL(ag_gemm, dim3(GRID), dim3(NTHREADS), LDS_BYTES, stream, p);
    HIP_CHECK(hipGetLastError());
}
```

```cpp
#include <hip/hip_runtime.h>
#include <rccl/rccl.h>
#include <cstdio>

#define HIP_CHECK(expr) do { hipError_t _e = (expr); if (_e != hipSuccess) { \
    fprintf(stderr, "HIP error %s at %s:%d\n", hipGetErrorString(_e), __FILE__, __LINE__); \
    return; } } while (0)

#define PERIOD10 56
#define OFFSET 150

#define PRIO_X 2
#define PRIO_W 2
#define PRIO_S 3
#define PRIO_M 0
#define SLEEP_H 1

#define SLEEP_M "1"
#define AG_YIELD

namespace {

constexpr int NR = 8;
constexpr int KDIM = 4096;
constexpr int MLOC = 512;
constexpr int NLOC = 2048;
constexpr int BM = 128, BN = 256, BK = 128;
constexpr int NSTEP = KDIM / BK;
constexpr int NGRP = MLOC / 32;
constexpr int NPIECE = NSTEP * NGRP;
constexpr int GRID = 256;
constexpr int NTHREADS = 960;

constexpr int A_STAGE = BM * BK;
constexpr int W_STAGE = BN * BK;
constexpr int LDS_A = 0;
constexpr int LDS_W = 3 * A_STAGE;
constexpr int LDS_CTRL = LDS_W + 3 * W_STAGE;
constexpr int LDS_LIST = LDS_CTRL + 128;
constexpr int LDS_DUMMY = LDS_LIST + 512;
constexpr int LDS_BYTES = LDS_DUMMY + 256;
constexpr int C_PROD = 0;
constexpr int C_WFULL = 4;
constexpr int C_BAD = 7;
constexpr int C_DONE = 8;
constexpr int C_CALL = 16;

constexpr int LIST_CAP = 64;
constexpr int PIECE = 4096;
constexpr unsigned INFO_RAW = 0x80u;

constexpr size_t WS_LAND = 0;
constexpr size_t WS_FLAG = WS_LAND + (size_t)2 * NR * NPIECE * PIECE;
constexpr size_t WS_CNT = WS_FLAG + (size_t)2 * NR * NPIECE * 8;
constexpr size_t WS_TMO = WS_CNT + GRID * 8;
constexpr size_t WS_STAMP = WS_TMO + 4096;
constexpr size_t WS_NEED = WS_STAMP + GRID * 32 * 8;

constexpr unsigned long long DEADLINE_TICKS = 100000000ull;

typedef unsigned u32x3 __attribute__((ext_vector_type(3)));
typedef unsigned u32x4 __attribute__((ext_vector_type(4)));
typedef int i32x2 __attribute__((ext_vector_type(2)));
typedef int i32x4 __attribute__((ext_vector_type(4)));
typedef int i32x8 __attribute__((ext_vector_type(8)));
typedef float f32x16 __attribute__((ext_vector_type(16)));
#define LAS __attribute__((address_space(3)))

struct Params {
    const unsigned char* x;
    const unsigned char* w;
    const float* sx;
    const float* sw;
    float* out;
    char* ws[NR];
    int rank;
    int pad;
};

__host__ __device__ __forceinline__ int src_offset(int b) { return ((b & 7) + 2 * ((b >> 3) & 3)) & (NR - 1); }

#define AG_CNT(r)
#define STAMP_T(t, i) do {} while (0)
#define PROF_DECL do {} while (0)
#define PROF(i) do {} while (0)
#define PROF_OUT(t, base) do {} while (0)

#define DMA_NOP 3
#define AG_STR2(x) #x
#define AG_STR(x) AG_STR2(x)
#define DMA_PAD "s_nop " AG_STR(DMA_NOP) "\n\t"
__device__ __forceinline__ void dma4(const void* base, unsigned v0, unsigned v1, unsigned v2, unsigned v3, unsigned lds) {
    unsigned keep;
    asm volatile("s_mov_b32 %0, m0\n\ts_mov_b32 m0, %6\n\ts_nop 0\n\t"
                 "global_load_lds_dwordx4 %1, %5\n\t" DMA_PAD
                 "global_load_lds_dwordx4 %2, %5 offset:1024\n\t" DMA_PAD
                 "global_load_lds_dwordx4 %3, %5 offset:2048\n\t" DMA_PAD
                 "global_load_lds_dwordx4 %4, %5 offset:3072\n\t"
                 "s_mov_b32 m0, %0"
                 : "=&s"(keep) : "v"(v0), "v"(v1), "v"(v2), "v"(v3), "s"(base), "s"(lds) : "memory");
}

__device__ __forceinline__ void ctrl_set(unsigned addr, unsigned val) {
    asm volatile("ds_write_b32 %0, %1" :: "v"(addr), "v"(val) : "memory");
}
__device__ __forceinline__ void ctrl_set1(unsigned addr, unsigned val) {
    asm volatile("s_mov_b64 exec, 1\n\tds_write_b32 %0, %1\n\ts_mov_b64 exec, -1" :: "v"(addr), "v"(val) : "memory");
}
__device__ __forceinline__ unsigned ctrl_get(unsigned addr) {
    unsigned v;
    asm volatile("ds_read_b32 %0, %1\n\ts_waitcnt lgkmcnt(0)" : "=v"(v) : "v"(addr) : "memory");
    return v;
}
template <int SL = 1>
__device__ __forceinline__ unsigned ctrl_wait(unsigned lane_addr, unsigned mask, int thr) {
    for (;;) {
        const unsigned v = ctrl_get(lane_addr);
        const unsigned ok = (unsigned)__builtin_amdgcn_ballot_w64((int)v > thr);
        if ((ok & mask) == mask) return v;
        __builtin_amdgcn_s_sleep(SL);
    }
}

struct XSet { u32x4 v0, v1, v2, v3; };

__device__ __forceinline__ unsigned xad(unsigned a, unsigned b, unsigned c) {
    unsigned d;
    asm("v_xad_u32 %0, %1, %2, %3" : "=v"(d) : "v"(a), "v"(b), "s"(c));
    return d;
}
__device__ __forceinline__ u32x4 unpack(unsigned c0, unsigned c1, unsigned c2, unsigned lo4, unsigned m60) {
    const unsigned r0 = c0 & m60, r1 = c1 & m60, r2 = c2 & m60;
    const unsigned lut = lo4 + 0x90801000u;
    u32x4 o;
    o[0] = xad(c0, r0, lo4);
    o[1] = xad(c1, r1, lo4);
    o[2] = xad(c2, r2, lo4);
    o[3] = (r0 >> 5) + (r1 >> 3) + __builtin_amdgcn_perm(lut, lut, r2 >> 5);
    return o;
}
__device__ __forceinline__ u32x3 pack(u32x4 x, unsigned lo4, unsigned& m) {
    unsigned f[4], z[4];
#pragma unroll
    for (int d = 0; d < 4; ++d) {
        const unsigned y = ((x[d] | 0x80808080u) - lo4) & 0x1F1F1F1Fu;
        z[d] = ((y + lo4) ^ x[d]) & 0x60606060u;
        f[d] = y | (x[d] & 0x80808080u);
    }
    m = (z[0] >> 5) | (z[1] >> 3) | (z[2] >> 1) | (z[3] << 1);
    u32x3 c;
    c[0] = f[0] | ((f[3] & 0x03030303u) << 5);
    c[1] = f[1] | ((f[3] & 0x0C0C0C0Cu) << 3);
    c[2] = f[2] | ((f[3] & 0x10101010u) << 1) | ((f[3] & 0x80808080u) >> 1);
    return c;
}

__device__ __forceinline__ void st3k(char* base, unsigned o, u32x4 a, u32x4 b, u32x4 c) {
    asm volatile("global_store_dwordx4 %0, %1, %4 sc0 sc1\n\t"
                 "global_store_dwordx4 %0, %2, %4 offset:1024 sc0 sc1\n\t"
                 "global_store_dwordx4 %0, %3, %4 offset:2048 sc0 sc1\n\ts_nop 1"
                 :: "v"(o), "v"(a), "v"(b), "v"(c), "s"(base) : "memory");
}
__device__ __forceinline__ void st1k(char* base, unsigned o, u32x4 a) {
    asm volatile("global_store_dwordx4 %0, %1, %2 offset:3072 sc0 sc1\n\ts_nop 1" :: "v"(o), "v"(a), "s"(base) : "memory");
}
__device__ __forceinline__ void st_ptr(char* p, u32x4 a) {
    asm volatile("global_store_dwordx4 %0, %1, off sc0 sc1\n\ts_nop 1" :: "v"(p), "v"(a) : "memory");
}

__device__ __forceinline__ float silu_out(float acc, float s) {
    const float y = acc * s;
    const float c = __builtin_amdgcn_fmed3f(y, -60.0f, 60.0f);
    const float e = __builtin_amdgcn_exp2f(c * -1.4426950408889634f);
    return y * __builtin_amdgcn_rcpf(1.0f + e);
}

__global__ void __launch_bounds__(NTHREADS, 4) ag_gemm(Params p) {
    extern __shared__ __attribute__((aligned(16))) char smem_[];
    LAS char* smem = (LAS char*)smem_;
    const unsigned lds0 = (unsigned)(size_t)smem;
    const int b = blockIdx.x;
    const int tid = threadIdx.x;
    const int lane = tid & 63;
    const int wave = __builtin_amdgcn_readfirstlane(tid >> 6);
    char* myws = p.ws[0];
    STAMP_T(0, 0);

    const unsigned src = (unsigned)(p.rank + src_offset(b)) & (NR - 1);
    const unsigned srcd = src;
    const int rb = (b >> 3) & 3, nt = b >> 5;
    const int n0 = nt * BN;
    const unsigned ctrl = lds0 + LDS_CTRL;
    const unsigned ctrl_lane = ctrl + (lane & 15) * 4;
    unsigned* tmo = (unsigned*)(myws + WS_TMO);

    if (wave < 8) {
        if (wave == 0) {
            ctrl_set(ctrl + (lane & 31) * 4, (lane & 24) == C_DONE ? 1u : 0u);
            asm volatile("s_waitcnt lgkmcnt(0)" ::: "memory");
        }
        __builtin_amdgcn_s_barrier();
        float sc = __builtin_bit_cast(float, __builtin_amdgcn_readfirstlane(__builtin_bit_cast(unsigned, p.sx[0] * p.sw[0])));
        float* outp = p.out;
        asm volatile("" : "+s"(sc), "+s"(outp));

        const int wm = wave >> 2, wn = wave & 3;
        f32x16 acc[2][2];
#pragma unroll
        for (int i = 0; i < 2; ++i)
#pragma unroll
            for (int j = 0; j < 2; ++j)
#pragma unroll
                for (int r = 0; r < 16; ++r) acc[i][j][r] = 0.f;

        const int h = lane >> 5;
        const int arow = 64 * wm + (lane & 31);
        const int a_rd = LDS_A + arow * 128 + ((((arow >> 1) & 7) ^ (2 * h)) << 4);
        const int q = (lane >> 1) & 7;
        const int w_rd = LDS_W + (32 * h + q) * 256 + 8 * (lane & 1) + (((4 * wn + ((lane >> 4) & 1)) ^ (2 * q)) << 4);
        const unsigned pa0 = lds0 + a_rd, pa1 = lds0 + (a_rd ^ 16), pa2 = lds0 + (a_rd ^ 64), pa3 = lds0 + (a_rd ^ 80);
        const unsigned pb0 = lds0 + w_rd, pb1 = lds0 + (w_rd ^ 32), pb2 = pb0 + 2 * W_STAGE, pb3 = pb1 + 2 * W_STAGE;
        const unsigned done_addr = lane == 0 ? ctrl + (C_DONE + wave) * 4 : lds0 + LDS_DUMMY + lane * 4;
        const unsigned pmask = 3u << (C_PROD + 2 * wm);
        unsigned cw, sn, st;
        __builtin_amdgcn_s_setprio(PRIO_M);
        asm volatile(
            ".macro AG_RDA t, lo, hi, off\n"
            "ds_read_b128 v[\\t:\\t+3], \\lo offset:\\off\n"
            "ds_read_b128 v[\\t+4:\\t+7], \\hi offset:\\off\n"
            ".endm\n"
            ".macro AG_RDB t, p, off\n"
            "ds_read_b64_tr_b8 v[\\t:\\t+1], \\p offset:\\off\n"
            "ds_read_b64_tr_b8 v[\\t+2:\\t+3], \\p offset:\\off+2048\n"
            "ds_read_b64_tr_b8 v[\\t+4:\\t+5], \\p offset:\\off+4096\n"
            "ds_read_b64_tr_b8 v[\\t+6:\\t+7], \\p offset:\\off+6144\n"
            ".endm\n"
            ".macro AG_MM c, a, b\n"
            "v_mfma_f32_32x32x64_f8f6f4 \\c, v[\\a:\\a+7], v[\\b:\\b+7], \\c cbsz:1 blgp:1\n"
            ".endm\n"
            ".macro AG_POLL mask\n"
            ".LAG_P\\@:\n"
            "ds_read_b32 %4, %15\n"
            "s_waitcnt lgkmcnt(0)\n"
            "v_cmp_lt_i32 vcc, %5, %4\n"
            "s_and_b32 %6, \\mask, vcc_lo\n"
            "s_cmp_eq_u32 %6, \\mask\n"
            "s_cbranch_scc1 .LAG_Q\\@\n"
            AG_CNT("s91")
            "s_sleep " SLEEP_M "\n"
            "s_branch .LAG_P\\@\n"
            ".LAG_Q\\@:\n"
            ".endm\n"
            ".macro AG_FETCH a0, a1, b0, b1, ao, q0, q1, wo\n"
            "AG_RDA \\a0, %7, %8, \\ao\n"
            "AG_RDB \\b0, \\q0, \\wo\n"
            "AG_RDA \\a1, %7, %8, \\ao+4096\n"
            "AG_RDB \\b1, \\q1, \\wo\n"
            ".endm\n"
            ".macro AG_HA a0, a1, b0, b1, f0, f1, ao, q0, q1, wo, look\n"
            "s_waitcnt lgkmcnt(6)\n"
            "AG_MM %0, \\a0, \\b0\n"
            "AG_RDA \\f0, %9, %10, \\ao\n"
            "AG_RDB \\f1, \\q0, \\wo+16384\n"
            "s_waitcnt lgkmcnt(10)\n"
            "AG_MM %1, \\a1, \\b0\n"
            "AG_RDA \\b0, %9, %10, \\ao+4096\n"
            ".if \\look\n"
            "ds_read_b32 %4, %15\n"
            ".endif\n"
            "s_waitcnt lgkmcnt(8+\\look)\n"
            "AG_MM %2, \\a0, \\b1\n"
            "AG_RDB \\a0, \\q1, \\wo+16384\n"
            "ds_add_u32 %16, %17\n"
            "AG_MM %3, \\a1, \\b1\n"
            "s_waitcnt lgkmcnt(5)\n"
            ".endm\n"
            ".macro AG_FOUR a0, a1, b0, b1\n"
            "AG_MM %0, \\a0, \\b0\n"
            "AG_MM %1, \\a1, \\b0\n"
            "s_waitcnt lgkmcnt(0)\n"
            "AG_MM %2, \\a0, \\b1\n"
            "AG_MM %3, \\a1, \\b1\n"
            ".endm\n"
            ".macro AG_STEP a0, a1, b0, b1, f0, f1, ao, q0, q1, wo, nao, nq0, nq1, nwo, mask\n"
            "s_add_u32 %5, %5, 1\n"
            "AG_HA \\a0, \\a1, \\b0, \\b1, \\f0, \\f1, \\ao, \\q0, \\q1, \\wo, 1\n"
            "v_cmp_lt_i32 vcc, %5, %4\n"
            "s_setprio 0\n"
            "s_and_b32 %6, %19, vcc_lo\n"
            "s_cbranch_scc0 .LAG_F\\@\n"
            "s_setprio 1\n"
            ".LAG_F\\@:\n"
            "s_and_b32 %6, \\mask, vcc_lo\n"
            "s_cmp_eq_u32 %6, \\mask\n"
            "s_cbranch_scc1 .LAG_R\\@\n"
            AG_CNT("s90")
            "AG_FOUR \\f0, \\b0, \\f1, \\a0\n"
            "AG_POLL \\mask\n"
            "AG_FETCH \\b1, \\f1, \\a1, \\f0, \\nao, \\nq0, \\nq1, \\nwo\n"
            "s_branch .LAG_E\\@\n"
            ".LAG_R\\@:\n"
            "AG_MM %0, \\f0, \\f1\n"
            "AG_RDA \\b1, %7, %8, \\nao\n"
            "AG_RDB \\a1, \\nq0, \\nwo\n"
            "AG_MM %1, \\b0, \\f1\n"
            "AG_RDA \\f1, %7, %8, \\nao+4096\n"
            "s_waitcnt lgkmcnt(9)\n"
            "AG_MM %2, \\f0, \\a0\n"
            "AG_RDB \\f0, \\nq1, \\nwo\n"
            "AG_MM %3, \\b0, \\a0\n"
            ".LAG_E\\@:\n"
            ".endm\n"
            "s_mov_b32 %5, 0\n"
            "AG_POLL %18\n"
            "AG_FETCH 80, 88, 96, 104, 0, %11, %12, 0\n"
            ".LAG_LOOP:\n"
            "AG_STEP 80, 88, 96, 104, 112, 120, 0, %11, %12, 0, 16384, %11, %12, 32768, %18\n"
            "s_cmp_eq_u32 %5, 31\n"
            "s_cbranch_scc1 .LAG_LAST\n"
            "AG_STEP 104, 120, 88, 112, 80, 96, 16384, %11, %12, 32768, 32768, %13, %14, 0, %18\n"
            "AG_STEP 112, 96, 120, 80, 104, 88, 32768, %13, %14, 0, 0, %11, %12, 0, %18\n"
            "s_branch .LAG_LOOP\n"
            ".LAG_LAST:\n"
            "AG_HA 104, 120, 88, 112, 80, 96, 16384, %11, %12, 32768, 0\n"
            "AG_FOUR 80, 88, 96, 104\n"
            "s_nop 15\n"
            "s_nop 3\n"
            : "+v"(acc[0][0]), "+v"(acc[1][0]), "+v"(acc[0][1]), "+v"(acc[1][1]), "=&v"(cw), "=&s"(sn), "=&s"(st)
            : "v"(pa0), "v"(pa1), "v"(pa2), "v"(pa3), "v"(pb0), "v"(pb1), "v"(pb2), "v"(pb3),
              "v"(ctrl_lane), "v"(done_addr), "v"(1u),
              "s"(pmask | (1u << C_WFULL)),
              "s"(1u << (C_DONE + (wave ^ 4)))
            : "memory", "vcc", "scc",
              "v80", "v81", "v82", "v83", "v84", "v85", "v86", "v87", "v88", "v89", "v90", "v91", "v92", "v93", "v94", "v95",
              "v96", "v97", "v98", "v99", "v100", "v101", "v102", "v103", "v104", "v105", "v106", "v107", "v108", "v109",
              "v110", "v111", "v112", "v113", "v114", "v115", "v116", "v117", "v118", "v119", "v120", "v121", "v122",
              "v123", "v124", "v125", "v126", "v127");
        static_assert(A_STAGE == 16384 && W_STAGE == 32768 && NSTEP == 32, "offsets in the K loop");
        const unsigned bad = __builtin_amdgcn_readlane(cw, C_BAD);
        STAMP_T(0, 9);

        if (bad) sc = __builtin_nanf("");
        const int m0 = (int)src * MLOC + rb * BM + wm * 64;
        const __amdgpu_buffer_rsrc_t ro = __builtin_amdgcn_make_buffer_rsrc(
            outp + (size_t)m0 * NLOC + n0 + wn * 64, 0, 64 * NLOC * 4, 0x00020000);
        const int vo = (4 * (lane >> 5)) * (NLOC * 4) + (lane & 31) * 4;
#pragma unroll
        for (int i = 0; i < 2; ++i)
#pragma unroll
            for (int r = 0; r < 16; ++r)
#pragma unroll
                for (int j = 0; j < 2; ++j)
                    __builtin_amdgcn_raw_buffer_store_b32(__builtin_bit_cast(unsigned, silu_out(acc[i][j][r], sc)), ro,
                                                          vo + j * 128, (32 * i + (r & 3) + 8 * (r >> 2)) * (NLOC * 4), 16);
    } else if (wave < 12) {
        const int pw = wave - 8;
        __builtin_amdgcn_s_barrier();
        __builtin_amdgcn_s_setprio(PRIO_X);
        unsigned call;
        while ((call = __builtin_amdgcn_readfirstlane(ctrl_get(ctrl + C_CALL * 4))) == 0u) __builtin_amdgcn_s_sleep(1);
        const unsigned par = call & 1u;
        const int grp = 4 * rb + pw;
        const unsigned long long* fl = (const unsigned long long*)(myws + WS_FLAG) + ((par * NR + srcd) * NGRP + grp) * NSTEP + (lane & 31);
        const __amdgpu_buffer_rsrc_t land = __builtin_amdgcn_make_buffer_rsrc(
            myws + WS_LAND + (size_t)(par * NR + srcd) * NPIECE * PIECE, 0, NPIECE * PIECE, 0x00020000);
        const unsigned prod_addr = ctrl + (C_PROD + pw) * 4;
        const unsigned free_mask = 0xFu << (C_DONE + 4 * (pw >> 1));
        const unsigned o = lane * 16, o3p = (lane >> 1) * 4;
        unsigned m60 = 0x60606060u;
        asm volatile("" : "+v"(m60));
        const unsigned prod_lane = lane == 0 ? prod_addr : lds0 + LDS_DUMMY + lane * 4;
        unsigned finfo = 0, ready = 0, rawm = 0;
        PROF_DECL;
        auto poll = [&](int step) -> bool {
            const unsigned long long t0 = __builtin_amdgcn_s_memrealtime();
            for (;;) {
                const unsigned long long v = __hip_atomic_load(fl, __ATOMIC_RELAXED, __HIP_MEMORY_SCOPE_SYSTEM);
                finfo = (unsigned)v;
                ready = (unsigned)__builtin_amdgcn_ballot_w64((unsigned)(v >> 32) >= call);
                rawm = (unsigned)__builtin_amdgcn_ballot_w64((finfo & INFO_RAW) != 0u);
                if ((ready >> step) & 1u) return true;
                if (__builtin_amdgcn_s_memrealtime() - t0 > 2000ull) {
                    if (__builtin_amdgcn_readfirstlane(__hip_atomic_load(tmo, __ATOMIC_RELAXED, __HIP_MEMORY_SCOPE_AGENT)) == call
                        || __builtin_amdgcn_s_memrealtime() - t0 > DEADLINE_TICKS) {
                        if (lane == 0) __hip_atomic_store(tmo, call, __ATOMIC_RELAXED, __HIP_MEMORY_SCOPE_AGENT);
                        return false;
                    }
                }
                __builtin_amdgcn_s_sleep(1);
            }
        };
        auto issue = [&](XSet& xs, int step) {
            const int so = (step * NGRP + grp) * PIECE;
            asm volatile("" ::: "memory");
            xs.v0 = __builtin_amdgcn_raw_buffer_load_b128(land, o, so, 17);
            xs.v1 = __builtin_amdgcn_raw_buffer_load_b128(land, o + 1024, so, 17);
            xs.v2 = __builtin_amdgcn_raw_buffer_load_b128(land, o + 2048, so, 17);
            xs.v3 = __builtin_amdgcn_raw_buffer_load_b128(land, (((rawm >> step) & 1u) ? o : o3p) + 3072, so, 17);
            asm volatile("" ::: "memory");
        };
        auto decode = [&](XSet& xs, int step, int slot) {
            const unsigned cw = *(volatile LAS unsigned*)(size_t)ctrl_lane;
            LAS char* a = smem + LDS_A + slot * A_STAGE + pw * PIECE;
            LAS char* d = a + lane * 16;
            u32x4 y0 = xs.v0, y1 = xs.v1, y2 = xs.v2, y3 = xs.v3;
            unsigned e = 0;
            if (!((rawm >> step) & 1u)) {
                const unsigned lo4 = __builtin_amdgcn_readlane(finfo, step) * 0x01010101u;
                e = xs.v3[0] >> (16 * (lane & 1));
#pragma unroll
                for (int i = 0; i < 4; ++i) {
                    const u32x4 y = unpack(xs.v0[i], xs.v1[i], xs.v2[i], lo4, m60);
                    y0[i] = y[0]; y1[i] = y[1]; y2[i] = y[2]; y3[i] = y[3];
                }
            }
            const unsigned pv = (e & 3u) << (5 + 2 * (e & 12u));
            const unsigned pa = (e & 3u) ? (unsigned)(size_t)a + ((e >> 2) & 0xFFCu) : lds0 + LDS_DUMMY + lane * 4;
            if (((unsigned)__builtin_amdgcn_ballot_w64((int)cw > step - 2) & free_mask) != free_mask)
                ctrl_wait<SLEEP_H>(ctrl_lane, free_mask, step - 2);
            asm volatile("" ::: "memory");
            *(LAS u32x4*)(d) = y0;
            *(LAS u32x4*)(d + 1024) = y1;
            *(LAS u32x4*)(d + 2048) = y2;
            *(LAS u32x4*)(d + 3072) = y3;
            asm volatile("ds_xor_b32 %0, %1\n\tds_add_u32 %2, %3" :: "v"(pa), "v"(pv), "v"(prod_lane), "v"(1u) : "memory");
        };
        XSet s0, s1;
        bool ok = poll(0);
        STAMP_T(512, 2);
        if (ok) {
            issue(s0, 0);
            int slot = 0;
            for (int s = 0; s < NSTEP - 2; s += 2) {
                if (!((ready >> (s + 1)) & 1u) && !(ok = poll(s + 1))) break;
                issue(s1, s + 1);
                decode(s0, s, slot);
                slot = slot == 2 ? 0 : slot + 1;
                if (!((ready >> (s + 2)) & 1u) && !(ok = poll(s + 2))) break;
                issue(s0, s + 2);
                decode(s1, s + 1, slot);
                slot = slot == 2 ? 0 : slot + 1;
            }
            if (ok && !((ready >> (NSTEP - 1)) & 1u)) ok = poll(NSTEP - 1);
            STAMP_T(512, 3);
            if (ok) {
                const unsigned tm = __hip_atomic_load(tmo, __ATOMIC_RELAXED, __HIP_MEMORY_SCOPE_AGENT);
                issue(s1, NSTEP - 1);
                decode(s0, NSTEP - 2, (NSTEP - 2) % 3);
                ok = __builtin_amdgcn_readfirstlane(tm) != call;
                if (ok) decode(s1, NSTEP - 1, (NSTEP - 1) % 3);
            }
        }
        if (!ok) {
            ctrl_set(ctrl + C_BAD * 4, 1u);
            ctrl_set(prod_addr, (unsigned)NSTEP);
        }
        STAMP_T(512, 8);
        PROF_OUT(512, 8);
    } else if (wave == 12) {
        const int wr = lane >> 4;
        const unsigned wo0 = (unsigned)(wr * NLOC + n0 + (((lane & 15) ^ (2 * wr)) << 4));
        const unsigned wo1 = (unsigned)((wr + 4) * NLOC + n0 + (((lane & 15) ^ (2 * (wr + 4))) << 4)) - 1024u;
        const unsigned wo2 = wo0 + 8 * NLOC - 2048u;
        const unsigned wo3 = wo1 + 8 * NLOC - 2048u;
        const unsigned full_addr = ctrl + C_WFULL * 4;
        __builtin_amdgcn_s_barrier();
        __builtin_amdgcn_s_setprio(PRIO_W);
        unsigned cw = 1;
        int slot = 0;
        for (int s = 0; s < NSTEP; ++s) {
            bool pend = s > 0;
            if (((unsigned)__builtin_amdgcn_ballot_w64((int)cw > s - 2) & (0xFFu << C_DONE)) != (0xFFu << C_DONE)) {
                asm volatile("s_waitcnt vmcnt(0)" ::: "memory");
                ctrl_set1(full_addr, (unsigned)s);
                pend = false;
                ctrl_wait<SLEEP_H>(ctrl_lane, 0xFFu << C_DONE, s - 2);
            }
            const unsigned char* g = p.w + (size_t)s * (BK * NLOC);
            const unsigned l = lds0 + LDS_W + slot * W_STAGE;
#pragma unroll
            for (int i = 0; i < 4; ++i) dma4(g + i * 16 * NLOC, wo0, wo1, wo2, wo3, l + i * 4096);
            if (pend) {
                asm volatile("s_waitcnt vmcnt(16)" ::: "memory");
                ctrl_set1(full_addr, (unsigned)s);
            }
#pragma unroll
            for (int i = 4; i < 7; ++i) dma4(g + i * 16 * NLOC, wo0, wo1, wo2, wo3, l + i * 4096);
            cw = *(volatile LAS unsigned*)(size_t)ctrl_lane;
            dma4(g + 7 * 16 * NLOC, wo0, wo1, wo2, wo3, l + 7 * 4096);
            slot = slot == 2 ? 0 : slot + 1;
        }
        asm volatile("s_waitcnt vmcnt(0)" ::: "memory");
        ctrl_set1(full_addr, (unsigned)NSTEP);
    } else {
        const unsigned long long tstart = __builtin_amdgcn_s_memrealtime();
        __builtin_amdgcn_s_setprio(PRIO_S);
        const int it = wave - 13;
        unsigned* cntp = (unsigned*)(myws + WS_CNT) + 2 * b + it;
        const unsigned cnt = __hip_atomic_load(cntp, __ATOMIC_RELAXED, __HIP_MEMORY_SCOPE_AGENT);
        const int grp = b & 15;
        const int step = (b >> 4) + 16 * it;
        const int lst_off = LDS_LIST + it * 256;
        const int r0 = lane >> 3;
        const unsigned char* xg = p.x + (size_t)(32 * grp + r0) * KDIM + step * BK;
        const int ca = ((lane & 7) ^ (r0 >> 1)) << 4, cb = ca ^ 64;
        const u32x4 x0 = *(const u32x4*)(xg + ca);
        const u32x4 x1 = *(const u32x4*)(xg + 8 * KDIM + cb);
        const u32x4 x2 = *(const u32x4*)(xg + 16 * KDIM + ca);
        const u32x4 x3 = *(const u32x4*)(xg + 24 * KDIM + cb);
        char* wsl = p.ws[lane >> 3];
        char* wsd[NR];
#pragma unroll
        for (int j = 0; j < NR; ++j) { wsd[j] = p.ws[j]; asm volatile("" : "+s"(wsd[j])); }
        asm volatile("" ::: "memory");
        __builtin_amdgcn_s_barrier();
        const unsigned call = __builtin_amdgcn_readfirstlane(cnt) + 1u;
        ctrl_set(ctrl + (C_CALL + it) * 4, call);
        const unsigned par = call & 1u;
        const unsigned o = lane * 16;
        {
            if (lane < 33) ctrl_set(lds0 + lst_off + lane * 4, 0u);
            unsigned sum = 0;
#pragma unroll
            for (int d = 0; d < 4; ++d) sum = __builtin_amdgcn_sad_u8(x0[d] & 0x7F7F7F7Fu, 0u, sum);
            sum += __builtin_amdgcn_update_dpp(0u, sum, 0x111, 0xf, 0xf, false);
            sum += __builtin_amdgcn_update_dpp(0u, sum, 0x112, 0xf, 0xf, false);
            sum += __builtin_amdgcn_update_dpp(0u, sum, 0x114, 0xf, 0xf, false);
            sum += __builtin_amdgcn_update_dpp(0u, sum, 0x118, 0xf, 0xf, false);
            const int tot = __builtin_amdgcn_readlane(sum, 15) + __builtin_amdgcn_readlane(sum, 31) +
                            __builtin_amdgcn_readlane(sum, 47) + __builtin_amdgcn_readlane(sum, 63);
            int lo = (tot - 20582) >> 10;
            lo = lo < 0 ? 0 : (lo > 96 ? 96 : lo);
            const unsigned lo4 = (unsigned)lo * 0x01010101u;
            unsigned m[4];
            u32x4 c0, c1, c2;
#pragma unroll
            for (int i = 0; i < 4; ++i) {
                const u32x3 c = pack((u32x4){x0[i], x1[i], x2[i], x3[i]}, lo4, m[i]);
                c0[i] = c[0]; c1[i] = c[1]; c2[i] = c[2];
            }
            unsigned n = 0;
#pragma unroll
            for (int v = 0; v < 4; ++v) n += __builtin_popcount((m[v] | (m[v] >> 1)) & 0x55555555u);
            if (n) {
                unsigned slot;
                asm volatile("ds_add_rtn_u32 %0, %1, %2\n\ts_waitcnt lgkmcnt(0)" : "=&v"(slot) : "v"(lds0 + lst_off + 128), "v"(n) : "memory");
#pragma unroll
                for (int v = 0; v < 4; ++v) {
                    unsigned mv = m[v];
                    while (mv != 0 && slot < LIST_CAP) {
                        const unsigned bit = (unsigned)__builtin_ctz(mv) & ~1u;
                        const unsigned dl = (mv >> bit) & 3u;
                        mv &= ~(3u << bit);
                        const unsigned pos = ((bit >> 1) & 3) * 1024 + lane * 16 + v * 4 + (bit >> 3);
                        *(LAS unsigned short*)(smem + lst_off + slot * 2) = (unsigned short)((pos << 2) | dl);
                        ++slot;
                    }
                }
            }
            asm volatile("" ::: "memory");
            const u32x4 lst = *(volatile LAS u32x4*)(smem + lst_off + (lane & 7) * 16);
            const bool raw = __builtin_amdgcn_readfirstlane(ctrl_get(lds0 + lst_off + 128)) > LIST_CAP;
            const u32x4 d0 = raw ? x0 : c0, d1 = raw ? x1 : c1, d2 = raw ? x2 : c2;
            const size_t off = WS_LAND + ((size_t)(par * NR + p.rank) * NPIECE + step * NGRP + grp) * PIECE;
            unsigned long long* fp = (unsigned long long*)(wsl + WS_FLAG) + ((par * NR + p.rank) * NGRP + grp) * NSTEP + step;
            const unsigned long long fv = ((unsigned long long)call << 32) | (raw ? INFO_RAW : (unsigned)lo);
            char* lp = wsl + off + 3072 + (lane & 7) * 16;
            STAMP_T(832, 12);

            const int ord = b + GRID * it;
            const unsigned long long due = tstart + (unsigned long long)(OFFSET + ord * PERIOD10 / 10);
            const bool early = __builtin_amdgcn_s_memrealtime() + 80 < due;
            if (early) {
                st3k(wsd[0] + off, o, d0, d1, d2);
                if (raw) st1k(wsd[0] + off, o, x3);
                else if (lane < 8) st_ptr(lp, lst);
                asm volatile("s_waitcnt vmcnt(0)" ::: "memory");
                if (lane == 0) __hip_atomic_store(fp, fv, __ATOMIC_RELAXED, __HIP_MEMORY_SCOPE_SYSTEM);
            }
            while (__builtin_amdgcn_s_memrealtime() < due) __builtin_amdgcn_s_sleep(1);
            STAMP_T(832, 14); STAMP_T(896, 15);
#pragma unroll
            for (int j = 1; j < NR; ++j) {
                st3k(wsd[j] + off, o, d0, d1, d2);
                if (raw) st1k(wsd[j] + off, o, x3);
            }
            if (!early) {
                st3k(wsd[0] + off, o, d0, d1, d2);
                if (raw) st1k(wsd[0] + off, o, x3);
            }
            if (!raw) st_ptr(lp, lst);
            asm volatile("s_waitcnt vmcnt(0)" ::: "memory");
            if ((lane & 7) == 0) __hip_atomic_store(fp, fv, __ATOMIC_RELAXED, __HIP_MEMORY_SCOPE_SYSTEM);
            STAMP_T(832, 1);
            STAMP_T(896, 13);
            if (lane == 0) __hip_atomic_store(cntp, call, __ATOMIC_RELAXED, __HIP_MEMORY_SCOPE_AGENT);
        }
    }
}

}

extern "C" void kernel_launch(int rank, int nranks, ncclComm_t comm,
                              hipStream_t stream,
                              void* const* d_in, const int* in_sizes, int n_in,
                              void* d_out, int out_size,
                              const int* in_dtypes, int out_dtype,
                              void* const* d_out_all,
                              void* const* d_ws_all, size_t ws_bytes) {
    (void)comm; (void)in_dtypes; (void)out_dtype; (void)d_out_all;
    if (nranks != NR || (unsigned)rank >= (unsigned)NR || n_in != 4 || ws_bytes < WS_NEED ||
        in_sizes[0] != MLOC * KDIM || in_sizes[1] != KDIM * NLOC || out_size != NR * MLOC * NLOC) {
        fprintf(stderr, "ag_gemm: unexpected problem shape\n");
        return;
    }
    HIP_CHECK(hipFuncSetAttribute((const void*)ag_gemm, hipFuncAttributeMaxDynamicSharedMemorySize, LDS_BYTES));
    Params p;
    p.x = (const unsigned char*)d_in[0];
    p.w = (const unsigned char*)d_in[1];
    p.sx = (const float*)d_in[2];
    p.sw = (const float*)d_in[3];
    p.out = (float*)d_out;
    for (int j = 0; j < NR; ++j) p.ws[j] = (char*)d_ws_all[(rank + j) % NR];
    p.rank = rank;
    p.pad = 0;
    hipLaunchKernelGGL(ag_gemm, dim3(GRID), dim3(NTHREADS), LDS_BYTES, stream, p);
    HIP_CHECK(hipGetLastError());
}
```

```cpp
#include <hip/hip_runtime.h>
#include <rccl/rccl.h>
#include <cstdio>

#define HIP_CHECK(expr) do { hipError_t _e = (expr); if (_e != hipSuccess) { \
    fprintf(stderr, "HIP error %s at %s:%d\n", hipGetErrorString(_e), __FILE__, __LINE__); \
    return; } } while (0)

#define PERIOD10 56
#define OFFSET 150
#define SEND_WIN 64
#define LW 4
#define SKEW_RB 0
#define SKEW_WM 0
#define OUT_AUX0 16
#define OUT_AUX1 16
#define PRIO_X 2
#define PRIO_W 2
#define PRIO_S 3
#define PRIO_M 0
#define SLEEP_H 1

#define SLEEP_M "1"
#define AG_YIELD

namespace {

constexpr int NR = 8;
constexpr int KDIM = 4096;
constexpr int MLOC = 512;
constexpr int NLOC = 2048;
constexpr int BM = 128, BN = 256, BK = 128;
constexpr int NSTEP = KDIM / BK;
constexpr int NGRP = MLOC / 32;
constexpr int NPIECE = NSTEP * NGRP;
constexpr int GRID = 256;
constexpr int NTHREADS = 1024;

constexpr int A_STAGE = BM * BK;
constexpr int W_STAGE = BN * BK;
constexpr int LDS_A = 0;
constexpr int LDS_W = 3 * A_STAGE;
constexpr int LDS_CTRL = LDS_W + 3 * W_STAGE;
constexpr int LDS_LIST = LDS_CTRL + 128;
constexpr int LDS_DUMMY = LDS_LIST + 5 * 256;
constexpr int LDS_INFO = LDS_DUMMY + 256;
constexpr int LDS_BYTES = LDS_INFO + 512;
constexpr int C_PROD = 0;
constexpr int C_WFULL = 4;
constexpr int C_BAD = 7;
constexpr int C_DONE = 8;

constexpr int LIST_CAP = 64;
constexpr int PIECE = 4096;
constexpr unsigned INFO_RAW = 0x80u, INFO_UP = 0x100u, INFO_SKIP = 0x200u;

constexpr size_t WS_LAND = 0;
constexpr size_t WS_FLAG = WS_LAND + (size_t)2 * NR * NPIECE * PIECE;
constexpr size_t WS_CNT = WS_FLAG + (size_t)2 * NR * NPIECE * 8;
constexpr size_t WS_TOK = WS_CNT + GRID * 32;
constexpr size_t WS_TMO = WS_TOK + NPIECE * 64;
constexpr size_t WS_STAMP = WS_TMO + 4096;
constexpr size_t WS_NEED = WS_STAMP + GRID * 36 * 8;

constexpr unsigned long long DEADLINE_TICKS = 100000000ull;

typedef unsigned u32x3 __attribute__((ext_vector_type(3)));
typedef unsigned u32x4 __attribute__((ext_vector_type(4)));
typedef int i32x2 __attribute__((ext_vector_type(2)));
typedef int i32x4 __attribute__((ext_vector_type(4)));
typedef int i32x8 __attribute__((ext_vector_type(8)));
typedef float f32x16 __attribute__((ext_vector_type(16)));
#define LAS __attribute__((address_space(3)))

struct Params {
    const unsigned char* x;
    const unsigned char* w;
    const float* sx;
    const float* sw;
    float* out;
    char* ws[NR];
    int rank;
    int pad;
};

#define NTX 2
__host__ __device__ __forceinline__ int row_block(int b) { return ((b & 7) / (8 / NTX)) * (32 / NTX) + (b >> 3) / NTX; }
__host__ __device__ __forceinline__ int col_tile(int b) { return ((b & 7) % (8 / NTX)) * NTX + (b >> 3) % NTX; }
__host__ __device__ __forceinline__ int src_offset(int b) { return row_block(b) & (NR - 1); }

#define AG_CNT(r)
#define STAMP_T(t, i) do {} while (0)
#define PROF_DECL do {} while (0)
#define PROF(i) do {} while (0)
#define PROF_OUT(t, base) do {} while (0)

#define DMA_NOP 3
#define AG_STR2(x) #x
#define AG_STR(x) AG_STR2(x)
#define DMA_PAD "s_nop " AG_STR(DMA_NOP) "\n\t"
__device__ __forceinline__ void dma4(const void* base, unsigned v0, unsigned v1, unsigned v2, unsigned v3, unsigned lds) {
    unsigned keep;
    asm volatile("s_mov_b32 %0, m0\n\ts_mov_b32 m0, %6\n\ts_nop 0\n\t"
                 "global_load_lds_dwordx4 %1, %5\n\t" DMA_PAD
                 "global_load_lds_dwordx4 %2, %5 offset:1024\n\t" DMA_PAD
                 "global_load_lds_dwordx4 %3, %5 offset:2048\n\t" DMA_PAD
                 "global_load_lds_dwordx4 %4, %5 offset:3072\n\t"
                 "s_mov_b32 m0, %0"
                 : "=&s"(keep) : "v"(v0), "v"(v1), "v"(v2), "v"(v3), "s"(base), "s"(lds) : "memory");
}

__device__ __forceinline__ void ctrl_set(unsigned addr, unsigned val) {
    asm volatile("ds_write_b32 %0, %1" :: "v"(addr), "v"(val) : "memory");
}
__device__ __forceinline__ void ctrl_set1(unsigned addr, unsigned val) {
    asm volatile("s_mov_b64 exec, 1\n\tds_write_b32 %0, %1\n\ts_mov_b64 exec, -1" :: "v"(addr), "v"(val) : "memory");
}
__device__ __forceinline__ unsigned ctrl_get(unsigned addr) {
    unsigned v;
    asm volatile("ds_read_b32 %0, %1\n\ts_waitcnt lgkmcnt(0)" : "=v"(v) : "v"(addr) : "memory");
    return v;
}
template <int SL = 1>
__device__ __forceinline__ unsigned ctrl_wait(unsigned lane_addr, unsigned mask, int thr) {
    for (;;) {
        const unsigned v = ctrl_get(lane_addr);
        const unsigned ok = (unsigned)__builtin_amdgcn_ballot_w64((int)v > thr);
        if ((ok & mask) == mask) return v;
        __builtin_amdgcn_s_sleep(SL);
    }
}

struct XSet { u32x4 v0, v1, v2, v3; };

__device__ __forceinline__ unsigned xad(unsigned a, unsigned b, unsigned c) {
    unsigned d;
    asm("v_xad_u32 %0, %1, %2, %3" : "=v"(d) : "v"(a), "v"(b), "s"(c));
    return d;
}
__device__ __forceinline__ u32x4 unpack(unsigned c0, unsigned c1, unsigned c2, unsigned lo4, unsigned m60) {
    const unsigned r0 = c0 & m60, r1 = c1 & m60, r2 = c2 & m60;
    const unsigned lut = lo4 + 0x90801000u;
    u32x4 o;
    o[0] = xad(c0, r0, lo4);
    o[1] = xad(c1, r1, lo4);
    o[2] = xad(c2, r2, lo4);
    o[3] = (r0 >> 5) + (r1 >> 3) + __builtin_amdgcn_perm(lut, lut, r2 >> 5);
    return o;
}
__device__ __forceinline__ u32x3 pack(u32x4 x, unsigned lo4, unsigned& m) {
    unsigned f[4], z[4];
#pragma unroll
    for (int d = 0; d < 4; ++d) {
        const unsigned y = ((x[d] | 0x80808080u) - lo4) & 0x1F1F1F1Fu;
        z[d] = ((y + lo4) ^ x[d]) & 0x60606060u;
        f[d] = y | (x[d] & 0x80808080u);
    }
    m = (z[0] >> 5) | (z[1] >> 3) | (z[2] >> 1) | (z[3] << 1);
    u32x3 c;
    c[0] = f[0] | ((f[3] & 0x03030303u) << 5);
    c[1] = f[1] | ((f[3] & 0x0C0C0C0Cu) << 3);
    c[2] = f[2] | ((f[3] & 0x10101010u) << 1) | ((f[3] & 0x80808080u) >> 1);
    return c;
}

__device__ __forceinline__ void st3k(char* base, unsigned o, u32x4 a, u32x4 b, u32x4 c) {
    asm volatile("global_store_dwordx4 %0, %1, %4 sc0 sc1\n\t"
                 "global_store_dwordx4 %0, %2, %4 offset:1024 sc0 sc1\n\t"
                 "global_store_dwordx4 %0, %3, %4 offset:2048 sc0 sc1\n\ts_nop 1"
                 :: "v"(o), "v"(a), "v"(b), "v"(c), "s"(base) : "memory");
}
__device__ __forceinline__ void st1k(char* base, unsigned o, u32x4 a) {
    asm volatile("global_store_dwordx4 %0, %1, %2 offset:3072 sc0 sc1\n\ts_nop 1" :: "v"(o), "v"(a), "s"(base) : "memory");
}
__device__ __forceinline__ void st_ptr(char* p, u32x4 a) {
    asm volatile("global_store_dwordx4 %0, %1, off sc0 sc1\n\ts_nop 1" :: "v"(p), "v"(a) : "memory");
}

__device__ __forceinline__ int piece_at(int ord, int lane) {
    int before = 0;
#pragma unroll
    for (int h = 0; h < 8; ++h) {
        const int t = lane + 1 - (SKEW_RB * (h >> 1) + SKEW_WM * (h & 1));
        before += 2 * (t < 0 ? 0 : (t > NSTEP ? NSTEP : t));
    }
    const int key = __builtin_popcountll(__builtin_amdgcn_ballot_w64(before <= ord));
    const int rem = ord - (key ? __builtin_amdgcn_readlane(before, key - 1) : 0);
    const int st = key - (SKEW_RB * (lane >> 2) + SKEW_WM * ((lane >> 1) & 1));
    const bool act = lane < NGRP && st >= 0 && st < NSTEP;
    const unsigned am = (unsigned)__builtin_amdgcn_ballot_w64(act);
    const int g = __builtin_ctz((unsigned)__builtin_amdgcn_ballot_w64(act && (int)__builtin_amdgcn_mbcnt_lo(am, 0u) == rem));
    return __builtin_amdgcn_readlane(st, g) * NGRP + g;
}

__device__ __forceinline__ float silu_out(float acc, float s) {
    const float y = acc * s;
    const float c = __builtin_amdgcn_fmed3f(y, -60.0f, 60.0f);
    const float e = __builtin_amdgcn_exp2f(c * -1.4426950408889634f);
    return y * __builtin_amdgcn_rcpf(1.0f + e);
}

__global__ void __launch_bounds__(NTHREADS, 4) ag_gemm(Params p) {
    extern __shared__ __attribute__((aligned(16))) char smem_[];
    LAS char* smem = (LAS char*)smem_;
    const unsigned lds0 = (unsigned)(size_t)smem;
    const int b = blockIdx.x;
    const int tid = threadIdx.x;
    const int lane = tid & 63;
    const int wave = __builtin_amdgcn_readfirstlane(tid >> 6);
    char* myws = p.ws[0];
    STAMP_T(0, 0);

    const unsigned src = (unsigned)(p.rank + src_offset(b)) & (NR - 1);
    const bool local = src_offset(b) == 0;
    const int rb = row_block(b) >> 3, nt = col_tile(b);
    const int n0 = nt * BN;
    const unsigned ctrl = lds0 + LDS_CTRL;
    const unsigned ctrl_lane = ctrl + (lane & 15) * 4;
    unsigned* tmo = (unsigned*)(myws + WS_TMO);

    if (wave < 8) {
        if (wave == 0) {
            ctrl_set(ctrl + (lane & 31) * 4, (lane & 24) == C_DONE ? 1u : 0u);
            ctrl_set(lds0 + LDS_INFO + lane * 4, 0u);
            ctrl_set(lds0 + LDS_INFO + 256 + lane * 4, 0u);
            asm volatile("s_waitcnt lgkmcnt(0)" ::: "memory");
        }
        __builtin_amdgcn_s_barrier();
        float sc = __builtin_bit_cast(float, __builtin_amdgcn_readfirstlane(__builtin_bit_cast(unsigned, p.sx[0] * p.sw[0])));
        float* outp = p.out;
        asm volatile("" : "+s"(sc), "+s"(outp));

        const int wm = wave >> 2, wn = wave & 3;
        f32x16 acc[2][2];
#pragma unroll
        for (int i = 0; i < 2; ++i)
#pragma unroll
            for (int j = 0; j < 2; ++j)
#pragma unroll
                for (int r = 0; r < 16; ++r) acc[i][j][r] = 0.f;

        const int h = lane >> 5;
        const int arow = 64 * wm + (lane & 31);
        const int a_rd = LDS_A + arow * 128 + ((((arow >> 1) & 7) ^ (2 * h)) << 4);
        const int q = (lane >> 1) & 7;
        const int w_rd = LDS_W + (32 * h + q) * 256 + 8 * (lane & 1) + (((4 * wn + ((lane >> 4) & 1)) ^ (2 * q)) << 4);
        const unsigned pa0 = lds0 + a_rd, pa1 = lds0 + (a_rd ^ 16), pa2 = lds0 + (a_rd ^ 64), pa3 = lds0 + (a_rd ^ 80);
        const unsigned pb0 = lds0 + w_rd, pb1 = lds0 + (w_rd ^ 32), pb2 = pb0 + 2 * W_STAGE, pb3 = pb1 + 2 * W_STAGE;
        const unsigned done_addr = lane == 0 ? ctrl + (C_DONE + wave) * 4 : lds0 + LDS_DUMMY + lane * 4;
        const unsigned pmask = local ? 0u : 3u << (C_PROD + 2 * wm);
        unsigned cw, sn, st;
        __builtin_amdgcn_s_setprio(PRIO_M);
        asm volatile(
            ".macro AG_RDA t, lo, hi, off\n"
            "ds_read_b128 v[\\t:\\t+3], \\lo offset:\\off\n"
            "ds_read_b128 v[\\t+4:\\t+7], \\hi offset:\\off\n"
            ".endm\n"
            ".macro AG_RDB t, p, off\n"
            "ds_read_b64_tr_b8 v[\\t:\\t+1], \\p offset:\\off\n"
            "ds_read_b64_tr_b8 v[\\t+2:\\t+3], \\p offset:\\off+2048\n"
            "ds_read_b64_tr_b8 v[\\t+4:\\t+5], \\p offset:\\off+4096\n"
            "ds_read_b64_tr_b8 v[\\t+6:\\t+7], \\p offset:\\off+6144\n"
            ".endm\n"
            ".macro AG_MM c, a, b\n"
            "v_mfma_f32_32x32x64_f8f6f4 \\c, v[\\a:\\a+7], v[\\b:\\b+7], \\c cbsz:1 blgp:1\n"
            ".endm\n"
            ".macro AG_POLL mask\n"
            ".LAG_P\\@:\n"
            "ds_read_b32 %4, %15\n"
            "s_waitcnt lgkmcnt(0)\n"
            "v_cmp_lt_i32 vcc, %5, %4\n"
            "s_and_b32 %6, \\mask, vcc_lo\n"
            "s_cmp_eq_u32 %6, \\mask\n"
            "s_cbranch_scc1 .LAG_Q\\@\n"
            AG_CNT("s91")
            "s_sleep " SLEEP_M "\n"
            "s_branch .LAG_P\\@\n"
            ".LAG_Q\\@:\n"
            ".endm\n"
            ".macro AG_FETCH a0, a1, b0, b1, ao, q0, q1, wo\n"
            "AG_RDA \\a0, %7, %8, \\ao\n"
            "AG_RDB \\b0, \\q0, \\wo\n"
            "AG_RDA \\a1, %7, %8, \\ao+4096\n"
            "AG_RDB \\b1, \\q1, \\wo\n"
            ".endm\n"
            ".macro AG_HA a0, a1, b0, b1, f0, f1, ao, q0, q1, wo, look\n"
            "s_waitcnt lgkmcnt(6)\n"
            "AG_MM %0, \\a0, \\b0\n"
            "AG_RDA \\f0, %9, %10, \\ao\n"
            "AG_RDB \\f1, \\q0, \\wo+16384\n"
            "s_waitcnt lgkmcnt(10)\n"
            "AG_MM %1, \\a1, \\b0\n"
            "AG_RDA \\b0, %9, %10, \\ao+4096\n"
            ".if \\look\n"
            "ds_read_b32 %4, %15\n"
            ".endif\n"
            "s_waitcnt lgkmcnt(8+\\look)\n"
            "AG_MM %2, \\a0, \\b1\n"
            "AG_RDB \\a0, \\q1, \\wo+16384\n"
            "ds_add_u32 %16, %17\n"
            "AG_MM %3, \\a1, \\b1\n"
            "s_waitcnt lgkmcnt(5)\n"
            ".endm\n"
            ".macro AG_FOUR a0, a1, b0, b1\n"
            "AG_MM %0, \\a0, \\b0\n"
            "AG_MM %1, \\a1, \\b0\n"
            "s_waitcnt lgkmcnt(0)\n"
            "AG_MM %2, \\a0, \\b1\n"
            "AG_MM %3, \\a1, \\b1\n"
            ".endm\n"
            ".macro AG_STEP a0, a1, b0, b1, f0, f1, ao, q0, q1, wo, nao, nq0, nq1, nwo, mask\n"
            "s_add_u32 %5, %5, 1\n"
            "AG_HA \\a0, \\a1, \\b0, \\b1, \\f0, \\f1, \\ao, \\q0, \\q1, \\wo, 1\n"
            "v_cmp_lt_i32 vcc, %5, %4\n"
            "s_setprio 0\n"
            "s_and_b32 %6, %21, vcc_lo\n"
            "s_cbranch_scc0 .LAG_F\\@\n"
            "s_setprio 1\n"
            ".LAG_F\\@:\n"
            "s_and_b32 %6, \\mask, vcc_lo\n"
            "s_cmp_eq_u32 %6, \\mask\n"
            "s_cbranch_scc1 .LAG_R\\@\n"
            AG_CNT("s90")
            "AG_FOUR \\f0, \\b0, \\f1, \\a0\n"
            "AG_POLL \\mask\n"
            "AG_FETCH \\b1, \\f1, \\a1, \\f0, \\nao, \\nq0, \\nq1, \\nwo\n"
            "s_branch .LAG_E\\@\n"
            ".LAG_R\\@:\n"
            "AG_MM %0, \\f0, \\f1\n"
            "AG_RDA \\b1, %7, %8, \\nao\n"
            "AG_RDB \\a1, \\nq0, \\nwo\n"
            "AG_MM %1, \\b0, \\f1\n"
            "AG_RDA \\f1, %7, %8, \\nao+4096\n"
            "s_waitcnt lgkmcnt(9)\n"
            "AG_MM %2, \\f0, \\a0\n"
            "AG_RDB \\f0, \\nq1, \\nwo\n"
            "AG_MM %3, \\b0, \\a0\n"
            ".LAG_E\\@:\n"
            ".endm\n"
            "s_mov_b32 %5, 0\n"
            "AG_POLL %18\n"
            "AG_FETCH 80, 88, 96, 104, 0, %11, %12, 0\n"
            ".LAG_LOOP:\n"
            "AG_STEP 80, 88, 96, 104, 112, 120, 0, %11, %12, 0, 16384, %11, %12, 32768, %19\n"
            "s_cmp_eq_u32 %5, 31\n"
            "s_cbranch_scc1 .LAG_LAST\n"
            "AG_STEP 104, 120, 88, 112, 80, 96, 16384, %11, %12, 32768, 32768, %13, %14, 0, %20\n"
            "AG_STEP 112, 96, 120, 80, 104, 88, 32768, %13, %14, 0, 0, %11, %12, 0, %18\n"
            "s_branch .LAG_LOOP\n"
            ".LAG_LAST:\n"
            "AG_HA 104, 120, 88, 112, 80, 96, 16384, %11, %12, 32768, 0\n"
            "AG_FOUR 80, 88, 96, 104\n"
            "s_nop 15\n"
            "s_nop 3\n"
            : "+v"(acc[0][0]), "+v"(acc[1][0]), "+v"(acc[0][1]), "+v"(acc[1][1]), "=&v"(cw), "=&s"(sn), "=&s"(st)
            : "v"(pa0), "v"(pa1), "v"(pa2), "v"(pa3), "v"(pb0), "v"(pb1), "v"(pb2), "v"(pb3),
              "v"(ctrl_lane), "v"(done_addr), "v"(1u),
              "s"(pmask | (1u << C_WFULL)), "s"(pmask | (2u << C_WFULL)), "s"(pmask | (4u << C_WFULL)),
              "s"(1u << (C_DONE + (wave ^ 4)))
            : "memory", "vcc", "scc",
              "v80", "v81", "v82", "v83", "v84", "v85", "v86", "v87", "v88", "v89", "v90", "v91", "v92", "v93", "v94", "v95",
              "v96", "v97", "v98", "v99", "v100", "v101", "v102", "v103", "v104", "v105", "v106", "v107", "v108", "v109",
              "v110", "v111", "v112", "v113", "v114", "v115", "v116", "v117", "v118", "v119", "v120", "v121", "v122",
              "v123", "v124", "v125", "v126", "v127");
        static_assert(A_STAGE == 16384 && W_STAGE == 32768 && NSTEP == 32, "offsets in the K loop");
        const unsigned bad = __builtin_amdgcn_readlane(cw, C_BAD);
        STAMP_T(0, 9);
        STAMP_T(448, 13);

        if (bad) sc = __builtin_nanf("");
        const int m0 = (int)src * MLOC + rb * BM + wm * 64;
        const __amdgpu_buffer_rsrc_t ro = __builtin_amdgcn_make_buffer_rsrc(
            outp + (size_t)m0 * NLOC + n0 + wn * 64, 0, 64 * NLOC * 4, 0x00020000);
        const int vo = (4 * (lane >> 5)) * (NLOC * 4) + (lane & 31) * 4;
#pragma unroll
        for (int r = 0; r < 16; ++r)
#pragma unroll
            for (int j = 0; j < 2; ++j)
                __builtin_amdgcn_raw_buffer_store_b32(__builtin_bit_cast(unsigned, silu_out(acc[0][j][r], sc)), ro,
                                                      vo + j * 128, ((r & 3) + 8 * (r >> 2)) * (NLOC * 4), OUT_AUX0);
#pragma unroll
        for (int r = 0; r < 16; ++r)
#pragma unroll
            for (int j = 0; j < 2; ++j)
                __builtin_amdgcn_raw_buffer_store_b32(__builtin_bit_cast(unsigned, silu_out(acc[1][j][r], sc)), ro,
                                                      vo + j * 128, (32 + (r & 3) + 8 * (r >> 2)) * (NLOC * 4), OUT_AUX1);
    } else if (wave < 13 - LW && local) {
        __builtin_amdgcn_s_barrier();
    } else if (wave < 12 && !local) {
        const int pw = wave - 8;
        unsigned* cntp = (unsigned*)(myws + WS_CNT) + 8 * b + pw;
        const unsigned cnt = __hip_atomic_load(cntp, __ATOMIC_RELAXED, __HIP_MEMORY_SCOPE_AGENT);
        __builtin_amdgcn_s_barrier();
        __builtin_amdgcn_s_setprio(PRIO_X);
        const unsigned call = __builtin_amdgcn_readfirstlane(cnt) + 1u;
        if (lane == 0) __hip_atomic_store(cntp, call, __ATOMIC_RELAXED, __HIP_MEMORY_SCOPE_AGENT);
        const unsigned par = call & 1u;
        const int grp = 4 * rb + pw;
        const unsigned info_addr = lds0 + LDS_INFO + (lane & 31) * 16 + pw * 4;
        const __amdgpu_buffer_rsrc_t land = __builtin_amdgcn_make_buffer_rsrc(
            myws + WS_LAND + (size_t)(par * NR + src) * NPIECE * PIECE, 0, NPIECE * PIECE, 0x00020000);
        const unsigned prod_addr = ctrl + (C_PROD + pw) * 4;
        const unsigned free_mask = 0xFu << (C_DONE + 4 * (pw >> 1));
        const unsigned o = lane * 16, o3p = (lane >> 1) * 4;
        unsigned m60 = 0x60606060u;
        asm volatile("" : "+v"(m60));
        const unsigned prod_lane = lane == 0 ? prod_addr : lds0 + LDS_DUMMY + lane * 4;
        unsigned finfo = 0, ready = 0, rawm = 0;
        PROF_DECL;
        auto look = [&]() {
            finfo = ctrl_get(info_addr);
            ready = (unsigned)__builtin_amdgcn_ballot_w64(finfo != 0u);
            rawm = (unsigned)__builtin_amdgcn_ballot_w64((finfo & INFO_RAW) != 0u);
            return __builtin_amdgcn_ballot_w64(finfo == INFO_SKIP) == 0;
        };
        auto poll = [&](int step) -> bool {
            for (;;) {
                if (!look()) return false;
                if ((ready >> step) & 1u) return true;
                __builtin_amdgcn_s_sleep(1);
            }
        };
        auto issue = [&](XSet& xs, int step) {
            const int so = (step * NGRP + grp) * PIECE;
            asm volatile("" ::: "memory");
            xs.v0 = __builtin_amdgcn_raw_buffer_load_b128(land, o, so, 17);
            xs.v1 = __builtin_amdgcn_raw_buffer_load_b128(land, o + 1024, so, 17);
            xs.v2 = __builtin_amdgcn_raw_buffer_load_b128(land, o + 2048, so, 17);
            xs.v3 = __builtin_amdgcn_raw_buffer_load_b128(land, (((rawm >> step) & 1u) ? o : o3p) + 3072, so, 17);
            asm volatile("" ::: "memory");
        };
        auto decode = [&](XSet& xs, int step, int slot) {
            const unsigned cw = *(volatile LAS unsigned*)(size_t)ctrl_lane;
            LAS char* a = smem + LDS_A + slot * A_STAGE + pw * PIECE;
            LAS char* d = a + lane * 16;
            u32x4 y0 = xs.v0, y1 = xs.v1, y2 = xs.v2, y3 = xs.v3;
            unsigned e = 0;
            if (!((rawm >> step) & 1u)) {
                const unsigned lo4 = (__builtin_amdgcn_readlane(finfo, step) & 0x7Fu) * 0x01010101u;
                e = xs.v3[0] >> (16 * (lane & 1));
#pragma unroll
                for (int i = 0; i < 4; ++i) {
                    const u32x4 y = unpack(xs.v0[i], xs.v1[i], xs.v2[i], lo4, m60);
                    y0[i] = y[0]; y1[i] = y[1]; y2[i] = y[2]; y3[i] = y[3];
                }
            }
            const unsigned pv = (e & 3u) << (5 + 2 * (e & 12u));
            const unsigned pa = (e & 3u) ? (unsigned)(size_t)a + ((e >> 2) & 0xFFCu) : lds0 + LDS_DUMMY + lane * 4;
            if (((unsigned)__builtin_amdgcn_ballot_w64((int)cw > step - 2) & free_mask) != free_mask)
                ctrl_wait<SLEEP_H>(ctrl_lane, free_mask, step - 2);
            asm volatile("" ::: "memory");
            *(LAS u32x4*)(d) = y0;
            *(LAS u32x4*)(d + 1024) = y1;
            *(LAS u32x4*)(d + 2048) = y2;
            *(LAS u32x4*)(d + 3072) = y3;
            asm volatile("ds_xor_b32 %0, %1\n\tds_add_u32 %2, %3" :: "v"(pa), "v"(pv), "v"(prod_lane), "v"(1u) : "memory");
        };
        XSet s0, s1;
        bool ok = poll(0);
        STAMP_T(512, 2);
        if (ok) {
            issue(s0, 0);
            int slot = 0;
            for (int s = 0;; s += 2) {
                if (!((ready >> (s + 1)) & 1u) && !(ok = look())) break;
                if ((ready >> (s + 1)) & 1u) {
                    issue(s1, s + 1);
                    decode(s0, s, slot);
                } else {
                    decode(s0, s, slot);
                    if (!(ok = poll(s + 1))) break;
                    issue(s1, s + 1);
                }
                slot = slot == 2 ? 0 : slot + 1;
                if (s + 2 == NSTEP) break;
                if (!((ready >> (s + 2)) & 1u) && !(ok = look())) break;
                if ((ready >> (s + 2)) & 1u) {
                    issue(s0, s + 2);
                    decode(s1, s + 1, slot);
                } else {
                    decode(s1, s + 1, slot);
                    if (!(ok = poll(s + 2))) break;
                    issue(s0, s + 2);
                }
                slot = slot == 2 ? 0 : slot + 1;
            }
            STAMP_T(512, 3);
            if (ok) decode(s1, NSTEP - 1, (NSTEP - 1) % 3);
        }
        if (!ok) ctrl_set(prod_addr, (unsigned)NSTEP);
        STAMP_T(512, 8);
        STAMP_T(704, 14);
    } else if (wave == 14 && !local) {
        unsigned* cntp = (unsigned*)(myws + WS_CNT) + 8 * b + 5;
        const unsigned cnt = __hip_atomic_load(cntp, __ATOMIC_RELAXED, __HIP_MEMORY_SCOPE_AGENT);
        __builtin_amdgcn_s_barrier();
        __builtin_amdgcn_s_setprio(PRIO_X);
        const unsigned call = __builtin_amdgcn_readfirstlane(cnt) + 1u;
        if (lane == 0) __hip_atomic_store(cntp, call, __ATOMIC_RELAXED, __HIP_MEMORY_SCOPE_AGENT);
        const unsigned long long* fl = (const unsigned long long*)(myws + WS_FLAG) + (((call & 1u) * NR + src) * 4 + rb) * (4 * NSTEP) + lane;
        const unsigned long long t0 = __builtin_amdgcn_s_memrealtime();
        for (int half = 0; half < 2; ++half) {
            unsigned long long seen0 = 0, seen1 = 0;
            for (;;) {
                const unsigned long long v0 = __hip_atomic_load(fl + 64 * half, __ATOMIC_RELAXED, __HIP_MEMORY_SCOPE_SYSTEM);
                unsigned long long v1 = 0;
                const bool both = half == 0 && (seen0 >> 40) != 0;
                if (both) v1 = __hip_atomic_load(fl + 64, __ATOMIC_RELAXED, __HIP_MEMORY_SCOPE_SYSTEM);
                const unsigned tm = __hip_atomic_load(tmo, __ATOMIC_RELAXED, __HIP_MEMORY_SCOPE_AGENT);
                if (__builtin_amdgcn_readfirstlane(tm) == call || __builtin_amdgcn_s_memrealtime() - t0 > DEADLINE_TICKS) {
                    if (lane == 0) __hip_atomic_store(tmo, call, __ATOMIC_RELAXED, __HIP_MEMORY_SCOPE_AGENT);
                    ctrl_set(ctrl + C_BAD * 4, 1u);
                    ctrl_set(lds0 + LDS_INFO + lane * 4, INFO_SKIP);
                    half = 2;
                    break;
                }
                const bool up0 = (unsigned)(v0 >> 32) >= call, up1 = (unsigned)(v1 >> 32) >= call;
                const unsigned long long m0 = __builtin_amdgcn_ballot_w64(up0), m1 = __builtin_amdgcn_ballot_w64(up1);
                if (m0 != seen0) ctrl_set(lds0 + LDS_INFO + half * 256 + lane * 4, up0 ? (unsigned)v0 | INFO_UP : 0u);
                if (m1 != seen1) ctrl_set(lds0 + LDS_INFO + 256 + lane * 4, up1 ? (unsigned)v1 | INFO_UP : 0u);
                seen0 = m0; seen1 = m1;
                if (m0 == ~0ull) break;
                __builtin_amdgcn_s_sleep(1);
            }
        }
    } else if (wave >= 12 && wave < 15) {
        const int ld = wave - 12, nld = local ? 3 : 2;
        PROF_DECL;
        const int wr = lane >> 4;
        const unsigned wo0 = (unsigned)(wr * NLOC + n0 + (((lane & 15) ^ (2 * wr)) << 4));
        const unsigned wo1 = (unsigned)((wr + 4) * NLOC + n0 + (((lane & 15) ^ (2 * (wr + 4))) << 4)) - 1024u;
        const unsigned wo2 = wo0 + 8 * NLOC - 2048u;
        const unsigned wo3 = wo1 + 8 * NLOC - 2048u;
        const unsigned xo0 = (lane >> 3) * KDIM + (((lane & 7) ^ (lane >> 4)) << 4);
        const unsigned xo1 = (xo0 ^ 64u) + 8 * KDIM - 1024, xo2 = xo0 + 16 * KDIM - 2048, xo3 = (xo0 ^ 64u) + 24 * KDIM - 3072;
        __builtin_amdgcn_s_barrier();
        __builtin_amdgcn_s_setprio(PRIO_W);
        for (int i = 0; i < ld; ++i) __builtin_amdgcn_s_sleep(16);
        for (int s = ld; s < NSTEP; s += nld) {
            const int sl = s % 3;
            ctrl_wait<SLEEP_H>(ctrl_lane, 0xFFu << C_DONE, s - 2);
            PROF(s < 6 ? 3 : 0);
            const unsigned char* g = p.w + (size_t)s * (BK * NLOC);
            const unsigned l = lds0 + LDS_W + sl * W_STAGE;
#pragma unroll
            for (int i = 0; i < 8; ++i) dma4(g + i * 16 * NLOC, wo0, wo1, wo2, wo3, l + i * 4096);
            if (local) {
                const unsigned char* gx = p.x + (size_t)(rb * BM) * KDIM + s * BK;
#pragma unroll
                for (int i = 0; i < 4; ++i) dma4(gx + i * 32 * KDIM, xo0, xo1, xo2, xo3, lds0 + LDS_A + sl * A_STAGE + i * PIECE);
            }
            PROF(s < 6 ? 3 : 1);
            asm volatile("s_waitcnt vmcnt(0)" ::: "memory");
            PROF(s < 6 ? 3 : 2);
            ctrl_set1(ctrl + (C_WFULL + sl) * 4, (unsigned)(s + 1));
        }
        PROF_OUT(768, 8);
    } else {
        const unsigned long long tstart = __builtin_amdgcn_s_memrealtime();
        __builtin_amdgcn_s_setprio(PRIO_S);
        const int sw = wave == 15 ? 4 : wave - 8;
        unsigned* cntp = (unsigned*)(myws + WS_CNT) + 8 * b + sw;
        const unsigned cnt = __hip_atomic_load(cntp, __ATOMIC_RELAXED, __HIP_MEMORY_SCOPE_AGENT);
        static_assert(NTX <= 4, "four local tiles per XCD");
        const int lq = (b >> 3) / (8 * NTX);
        const int r = ((b >> 3) - NTX * (1 + lq)) * 8 + (b & 7);
        int ord = local ? NPIECE - 32 * (5 - sw) + (lq * NTX + (b >> 3) % NTX) * 8 + (b & 7) : (r >= 192 ? r - 192 : r + 32);
        const int ord_end = local ? NPIECE : NPIECE - 32 * LW;
        const int lst_off = LDS_LIST + sw * 256;
        const int ca = (lane >> 3) * KDIM + (((lane & 7) ^ (lane >> 4)) << 4), cb = ca ^ 64;
        u32x4 x0, x1, x2, x3;
        int piece;
        auto load_x = [&]() {
            piece = piece_at(ord, lane);
            const unsigned char* xg = p.x + (size_t)(piece & 15) * (32 * KDIM) + (piece >> 4) * BK;
            x0 = *(const u32x4*)(xg + ca);
            x1 = *(const u32x4*)(xg + 8 * KDIM + cb);
            x2 = *(const u32x4*)(xg + 16 * KDIM + ca);
            x3 = *(const u32x4*)(xg + 24 * KDIM + cb);
        };
        load_x();
        char* wsl = p.ws[lane >> 3];
        char* wsd[NR];
#pragma unroll
        for (int j = 1; j < NR; ++j) { wsd[j] = p.ws[j]; asm volatile("" : "+s"(wsd[j])); }
        asm volatile("" ::: "memory");
        __builtin_amdgcn_s_barrier();
        const unsigned call = __builtin_amdgcn_readfirstlane(cnt) + 1u;
        const unsigned par = call & 1u;
        const unsigned o = lane * 16;
        unsigned* tok = (unsigned*)(myws + WS_TOK);
        for (;;) {
            const int step = piece >> 4, grp = piece & 15;
            if (lane < 33) ctrl_set(lds0 + lst_off + lane * 4, 0u);
            unsigned sum = 0;
#pragma unroll
            for (int d = 0; d < 4; ++d) sum = __builtin_amdgcn_sad_u8(x0[d] & 0x7F7F7F7Fu, 0u, sum);
            sum += __builtin_amdgcn_update_dpp(0u, sum, 0x111, 0xf, 0xf, false);
            sum += __builtin_amdgcn_update_dpp(0u, sum, 0x112, 0xf, 0xf, false);
            sum += __builtin_amdgcn_update_dpp(0u, sum, 0x114, 0xf, 0xf, false);
            sum += __builtin_amdgcn_update_dpp(0u, sum, 0x118, 0xf, 0xf, false);
            const int tot = __builtin_amdgcn_readlane(sum, 15) + __builtin_amdgcn_readlane(sum, 31) +
                            __builtin_amdgcn_readlane(sum, 47) + __builtin_amdgcn_readlane(sum, 63);
            int lo = (tot - 20582) >> 10;
            lo = lo < 0 ? 0 : (lo > 96 ? 96 : lo);
            const unsigned lo4 = (unsigned)lo * 0x01010101u;
            unsigned m[4];
            u32x4 c0, c1, c2;
#pragma unroll
            for (int i = 0; i < 4; ++i) {
                const u32x3 c = pack((u32x4){x0[i], x1[i], x2[i], x3[i]}, lo4, m[i]);
                c0[i] = c[0]; c1[i] = c[1]; c2[i] = c[2];
            }
            unsigned n = 0;
#pragma unroll
            for (int v = 0; v < 4; ++v) n += __builtin_popcount((m[v] | (m[v] >> 1)) & 0x55555555u);
            if (n) {
                unsigned slot;
                asm volatile("ds_add_rtn_u32 %0, %1, %2\n\ts_waitcnt lgkmcnt(0)" : "=&v"(slot) : "v"(lds0 + lst_off + 128), "v"(n) : "memory");
#pragma unroll
                for (int v = 0; v < 4; ++v) {
                    unsigned mv = m[v];
                    while (mv != 0 && slot < LIST_CAP) {
                        const unsigned bit = (unsigned)__builtin_ctz(mv) & ~1u;
                        const unsigned dl = (mv >> bit) & 3u;
                        mv &= ~(3u << bit);
                        const unsigned pos = ((bit >> 1) & 3) * 1024 + lane * 16 + v * 4 + (bit >> 3);
                        *(LAS unsigned short*)(smem + lst_off + slot * 2) = (unsigned short)((pos << 2) | dl);
                        ++slot;
                    }
                }
            }
            asm volatile("" ::: "memory");
            const u32x4 lst = *(volatile LAS u32x4*)(smem + lst_off + (lane & 7) * 16);
            const bool raw = __builtin_amdgcn_readfirstlane(ctrl_get(lds0 + lst_off + 128)) > LIST_CAP;
            u32x4 d0 = raw ? x0 : c0, d1 = raw ? x1 : c1, d2 = raw ? x2 : c2;
            const size_t off = WS_LAND + ((size_t)(par * NR + p.rank) * NPIECE + step * NGRP + grp) * PIECE;
            unsigned long long* fp = (unsigned long long*)(wsl + WS_FLAG) + (((par * NR + p.rank) * 4 + (grp >> 2)) * NSTEP + step) * 4 + (grp & 3);
            char* lp = wsl + off + 3072 + (lane & 7) * 16;
            asm volatile("" : "+v"(d0), "+v"(d1), "+v"(d2), "+v"(lp));
            if (ord < 32) STAMP_T(960, 12);
            const unsigned long long due = tstart + (unsigned long long)(OFFSET + ord * PERIOD10 / 10);
            while (__builtin_amdgcn_s_memrealtime() + 60 < due) __builtin_amdgcn_s_sleep(2);
            if (ord >= SEND_WIN && ord < NPIECE - 32 * (LW - 1))
                while (__builtin_amdgcn_readfirstlane(__hip_atomic_load(tok + (ord - SEND_WIN) * 16, __ATOMIC_RELAXED, __HIP_MEMORY_SCOPE_AGENT)) < call
                       && __builtin_amdgcn_s_memrealtime() < due + 5000) __builtin_amdgcn_s_sleep(1);
            while (__builtin_amdgcn_s_memrealtime() < due) __builtin_amdgcn_s_sleep(1);
#pragma unroll
            for (int j = 1; j < NR; ++j) {
                st3k(wsd[j] + off, o, d0, d1, d2);
                if (raw) st1k(wsd[j] + off, o, x3);
            }
            if (!raw && lane >= 8) st_ptr(lp, lst);
            asm volatile("s_waitcnt vmcnt(0)" ::: "memory");
            if (lane >= 8 && (lane & 7) == 0)
                __hip_atomic_store(fp, ((unsigned long long)call << 32) | (raw ? INFO_RAW : (unsigned)lo),
                                   __ATOMIC_RELAXED, __HIP_MEMORY_SCOPE_SYSTEM);
            if (lane == 0) __hip_atomic_store(tok + ord * 16, call, __ATOMIC_RELAXED, __HIP_MEMORY_SCOPE_AGENT);
            ord += 224;
            if (ord >= ord_end) break;
            load_x();
        }
        if (lane == 0) __hip_atomic_store(cntp, call, __ATOMIC_RELAXED, __HIP_MEMORY_SCOPE_AGENT);
    }
}

}

extern "C" void kernel_launch(int rank, int nranks, ncclComm_t comm,
                              hipStream_t stream,
                              void* const* d_in, const int* in_sizes, int n_in,
                              void* d_out, int out_size,
                              const int* in_dtypes, int out_dtype,
                              void* const* d_out_all,
                              void* const* d_ws_all, size_t ws_bytes) {
    (void)comm; (void)in_dtypes; (void)out_dtype; (void)d_out_all;
    if (nranks != NR || (unsigned)rank >= (unsigned)NR || n_in != 4 || ws_bytes < WS_NEED ||
        in_sizes[0] != MLOC * KDIM || in_sizes[1] != KDIM * NLOC || out_size != NR * MLOC * NLOC) {
        fprintf(stderr, "ag_gemm: unexpected problem shape\n");
        return;
    }
    HIP_CHECK(hipFuncSetAttribute((const void*)ag_gemm, hipFuncAttributeMaxDynamicSharedMemorySize, LDS_BYTES));
    Params p;
    p.x = (const unsigned char*)d_in[0];
    p.w = (const unsigned char*)d_in[1];
    p.sx = (const float*)d_in[2];
    p.sw = (const float*)d_in[3];
    p.out = (float*)d_out;
    for (int j = 0; j < NR; ++j) p.ws[j] = (char*)d_ws_all[(rank + j) % NR];
    p.rank = rank;
    p.pad = 0;
    hipLaunchKernelGGL(ag_gemm, dim3(GRID), dim3(NTHREADS), LDS_BYTES, stream, p);
    HIP_CHECK(hipGetLastError());
}
```

```cpp
#include <hip/hip_runtime.h>
#include <rccl/rccl.h>
#include <cstdio>

#define HIP_CHECK(expr) do { hipError_t _e = (expr); if (_e != hipSuccess) { \
    fprintf(stderr, "HIP error %s at %s:%d\n", hipGetErrorString(_e), __FILE__, __LINE__); \
    return; } } while (0)

#define PERIOD10 57
#define OFFSET 40
#define SEND_WIN 72
#define LW 4
#define SKEW 0x0000
#define OUT_AUX0 16
#define OUT_AUX1 16
#define RAW_FIRST 16
#define SLEEP_P 1
#define HOLD_LEAD 25
#define PRIO_X 2
#define PRIO_W 2
#define PRIO_S 3
#define PRIO_M 0
#define SLEEP_H 1

#define SLEEP_M "1"
#define AG_YIELD

namespace {

constexpr int NR = 8;
constexpr int KDIM = 4096;
constexpr int MLOC = 512;
constexpr int NLOC = 2048;
constexpr int BM = 128, BN = 256, BK = 128;
constexpr int NSTEP = KDIM / BK;
constexpr int NGRP = MLOC / 32;
constexpr int NPIECE = NSTEP * NGRP;
constexpr int GRID = 256;
constexpr int NTHREADS = 1024;

constexpr int A_STAGE = BM * BK;
constexpr int W_STAGE = BN * BK;
constexpr int LDS_A = 0;
constexpr int LDS_W = 3 * A_STAGE;
constexpr int LDS_CTRL = LDS_W + 3 * W_STAGE;
constexpr int LDS_LIST = LDS_CTRL + 128;
constexpr int LDS_DUMMY = LDS_LIST + 5 * 256;
constexpr int LDS_INFO = LDS_DUMMY + 256;
constexpr int LDS_BYTES = LDS_INFO + 512;
constexpr int C_PROD = 0;
constexpr int C_WFULL = 4;
constexpr int C_BAD = 7;
constexpr int C_HOLD = 16;
constexpr int C_SENT = 0;
constexpr int C_DONE = 8;

constexpr int LIST_CAP = 64;
constexpr int PIECE = 4096;
constexpr unsigned INFO_RAW = 0x80u, INFO_UP = 0x100u, INFO_SKIP = 0x200u;

constexpr size_t LAND_BYTES = (size_t)NPIECE * PIECE, FLAG_BYTES = (size_t)NPIECE * 8;
constexpr size_t SRC_FLAG = 2 * LAND_BYTES, SRC_BYTES = SRC_FLAG + 65536;
constexpr size_t WS_CNT = NR * SRC_BYTES;
constexpr size_t WS_TOK = WS_CNT + GRID * 32;
constexpr size_t WS_LATE = WS_TOK + NPIECE * 64;
constexpr size_t WS_TMO = WS_LATE + 256;
constexpr size_t WS_STAMP = WS_TMO + 4096;
constexpr size_t WS_NEED = WS_STAMP + GRID * 36 * 8;

constexpr unsigned long long DEADLINE_TICKS = 100000000ull;

typedef unsigned u32x3 __attribute__((ext_vector_type(3)));
typedef unsigned u32x4 __attribute__((ext_vector_type(4)));
typedef int i32x2 __attribute__((ext_vector_type(2)));
typedef int i32x4 __attribute__((ext_vector_type(4)));
typedef int i32x8 __attribute__((ext_vector_type(8)));
typedef float f32x16 __attribute__((ext_vector_type(16)));
#define LAS __attribute__((address_space(3)))

struct Params {
    const unsigned char* x;
    const unsigned char* w;
    const float* sx;
    const float* sw;
    float* out;
    char* ws[NR];
    int rank;
    int pad;
};

#define NTX 2
__host__ __device__ __forceinline__ int row_block(int b) { return ((b & 7) / (8 / NTX)) * (32 / NTX) + (b >> 3) / NTX; }
__host__ __device__ __forceinline__ int col_tile(int b) { return ((b & 7) % (8 / NTX)) * NTX + (b >> 3) % NTX; }
__host__ __device__ __forceinline__ int src_offset(int b) { return row_block(b) & (NR - 1); }

#define AG_CNT(r)
#define STAMP_T(t, i) do {} while (0)
#define PROF_DECL do {} while (0)
#define PROF(i) do {} while (0)
#define PROF_OUT(t, base) do {} while (0)

#define DMA_NOP 3
#define AG_STR2(x) #x
#define AG_STR(x) AG_STR2(x)
#define DMA_PAD "s_nop " AG_STR(DMA_NOP) "\n\t"
__device__ __forceinline__ void dma4(const void* base, unsigned v0, unsigned v1, unsigned v2, unsigned v3, unsigned lds) {
    unsigned keep;
    asm volatile("s_mov_b32 %0, m0\n\ts_mov_b32 m0, %6\n\ts_nop 0\n\t"
                 "global_load_lds_dwordx4 %1, %5\n\t" DMA_PAD
                 "global_load_lds_dwordx4 %2, %5 offset:1024\n\t" DMA_PAD
                 "global_load_lds_dwordx4 %3, %5 offset:2048\n\t" DMA_PAD
                 "global_load_lds_dwordx4 %4, %5 offset:3072\n\t"
                 "s_mov_b32 m0, %0"
                 : "=&s"(keep) : "v"(v0), "v"(v1), "v"(v2), "v"(v3), "s"(base), "s"(lds) : "memory");
}

__device__ __forceinline__ void ctrl_set(unsigned addr, unsigned val) {
    asm volatile("ds_write_b32 %0, %1" :: "v"(addr), "v"(val) : "memory");
}
__device__ __forceinline__ void ctrl_set1(unsigned addr, unsigned val) {
    asm volatile("s_mov_b64 exec, 1\n\tds_write_b32 %0, %1\n\ts_mov_b64 exec, -1" :: "v"(addr), "v"(val) : "memory");
}
__device__ __forceinline__ unsigned ctrl_get(unsigned addr) {
    unsigned v;
    asm volatile("ds_read_b32 %0, %1\n\ts_waitcnt lgkmcnt(0)" : "=v"(v) : "v"(addr) : "memory");
    return v;
}
template <int SL = 1>
__device__ __forceinline__ unsigned ctrl_wait(unsigned lane_addr, unsigned mask, int thr) {
    for (;;) {
        const unsigned v = ctrl_get(lane_addr);
        const unsigned ok = (unsigned)__builtin_amdgcn_ballot_w64((int)v > thr);
        if ((ok & mask) == mask) return v;
        __builtin_amdgcn_s_sleep(SL);
    }
}

__device__ __forceinline__ u32x3 pack(u32x4 x, unsigned lo4, unsigned& m) {
    unsigned f[4], z[4];
#pragma unroll
    for (int d = 0; d < 4; ++d) {
        const unsigned y = ((x[d] | 0x80808080u) - lo4) & 0x1F1F1F1Fu;
        z[d] = ((y + lo4) ^ x[d]) & 0x60606060u;
        f[d] = y | (x[d] & 0x80808080u);
    }
    m = (z[0] >> 5) | (z[1] >> 3) | (z[2] >> 1) | (z[3] << 1);
    u32x3 c;
    c[0] = f[0] | ((f[3] & 0x03030303u) << 5);
    c[1] = f[1] | ((f[3] & 0x0C0C0C0Cu) << 3);
    c[2] = f[2] | ((f[3] & 0x10101010u) << 1) | ((f[3] & 0x80808080u) >> 1);
    return c;
}

__device__ __forceinline__ void st3k(char* base, unsigned o, u32x4 a, u32x4 b, u32x4 c) {
    asm volatile("global_store_dwordx4 %0, %1, %4 sc0 sc1\n\t"
                 "global_store_dwordx4 %0, %2, %4 offset:1024 sc0 sc1\n\t"
                 "global_store_dwordx4 %0, %3, %4 offset:2048 sc0 sc1\n\ts_nop 1"
                 :: "v"(o), "v"(a), "v"(b), "v"(c), "s"(base) : "memory");
}
__device__ __forceinline__ void st1k(char* base, unsigned o, u32x4 a) {
    asm volatile("global_store_dwordx4 %0, %1, %2 offset:3072 sc0 sc1\n\ts_nop 1" :: "v"(o), "v"(a), "s"(base) : "memory");
}
__device__ __forceinline__ void st_ptr(char* p, u32x4 a) {
    asm volatile("global_store_dwordx4 %0, %1, off sc0 sc1\n\ts_nop 1" :: "v"(p), "v"(a) : "memory");
}

__device__ __forceinline__ int piece_at(int ord, int lane) {
    if (SKEW == 0) return ord;
    int before = 0;
#pragma unroll
    for (int h = 0; h < 4; ++h) {
        const int t = lane + 1 - ((SKEW >> (4 * h)) & 15);
        before += 4 * (t < 0 ? 0 : (t > NSTEP ? NSTEP : t));
    }
    const int key = __builtin_popcountll(__builtin_amdgcn_ballot_w64(before <= ord));
    const int rem = ord - (key ? __builtin_amdgcn_readlane(before, key - 1) : 0);
    const int st = key - ((SKEW >> (lane & 12)) & 15);
    const bool act = lane < NGRP && st >= 0 && st < NSTEP;
    const unsigned am = (unsigned)__builtin_amdgcn_ballot_w64(act);
    const int g = __builtin_ctz((unsigned)__builtin_amdgcn_ballot_w64(act && (int)__builtin_amdgcn_mbcnt_lo(am, 0u) == rem));
    return __builtin_amdgcn_readlane(st, g) * NGRP + g;
}

__device__ __forceinline__ float silu_out(float acc, float s) {
    const float y = acc * s;
    const float c = __builtin_amdgcn_fmed3f(y, -60.0f, 60.0f);
    const float e = __builtin_amdgcn_exp2f(c * -1.4426950408889634f);
    return y * __builtin_amdgcn_rcpf(1.0f + e);
}

__global__ void __launch_bounds__(NTHREADS, 4) ag_gemm(Params p) {
    extern __shared__ __attribute__((aligned(16))) char smem_[];
    LAS char* smem = (LAS char*)smem_;
    const unsigned lds0 = (unsigned)(size_t)smem;
    const int b = blockIdx.x & (GRID - 1);
    const int tid = (threadIdx.x + NTHREADS - 64) & (NTHREADS - 1);
    const int lane = tid & 63;
    const int wave = __builtin_amdgcn_readfirstlane(tid >> 6);
    asm volatile("" :: "s"(p.x), "s"(p.w), "s"(p.ws[0]), "s"(p.ws[1]), "s"(p.ws[2]), "s"(p.ws[3]), "s"(p.ws[4]),
                 "s"(p.ws[5]), "s"(p.ws[6]), "s"(p.ws[7]), "s"(p.rank));
    char* myws = p.ws[0];
    STAMP_T(960, 0);

    const unsigned src = (unsigned)(p.rank + src_offset(b)) & (NR - 1);
    const bool local = src_offset(b) == 0;
    const int rb = row_block(b) >> 3, nt = col_tile(b);
    const int n0 = nt * BN;
    static_assert(NTX <= 4, "four local tiles per XCD");
    const int lq = (b >> 3) / (8 * NTX);
    const int rr = ((b >> 3) - NTX * (1 + lq)) * 8 + (b & 7);
    const int ord0 = local ? NPIECE - 32 * LW + (lq * NTX + (b >> 3) % NTX) * 8 + (b & 7) : rr;
    const int ord_end = local ? NPIECE : NPIECE - 32 * LW;
    const unsigned ctrl = lds0 + LDS_CTRL;
    const unsigned ctrl_lane = ctrl + (lane & 15) * 4;
    unsigned* tmo = (unsigned*)(myws + WS_TMO);

    if (wave < 8) {
        if (wave == 1 && (b >> 3) == 12 && lane >= 8 && (lane & 7) < 3) {
            const char* t = p.ws[lane >> 3] + (lane & 7) * LAND_BYTES + (b & 7) * 128;
            __builtin_amdgcn_global_load_lds((const __attribute__((address_space(1))) void*)t, (LAS void*)(smem + LDS_DUMMY), 4, 0, 17);
        }
        if (wave == 0) {
            ctrl_set(ctrl + (lane & 31) * 4, (lane & 24) == C_DONE ? 1u : 0u);
            ctrl_set(lds0 + LDS_INFO + lane * 4, 0u);
            ctrl_set(lds0 + LDS_INFO + 256 + lane * 4, 0u);
            asm volatile("s_waitcnt lgkmcnt(0)" ::: "memory");
        }
        __builtin_amdgcn_s_barrier();
        float sc = __builtin_bit_cast(float, __builtin_amdgcn_readfirstlane(__builtin_bit_cast(unsigned, p.sx[0] * p.sw[0])));
        float* outp = p.out;
        asm volatile("" : "+s"(sc), "+s"(outp));

        const int wm = wave >> 2, wn = wave & 3;
        f32x16 acc[2][2];
#pragma unroll
        for (int i = 0; i < 2; ++i)
#pragma unroll
            for (int j = 0; j < 2; ++j)
#pragma unroll
                for (int r = 0; r < 16; ++r) acc[i][j][r] = 0.f;

        const int h = lane >> 5;
        const int arow = 64 * wm + (lane & 31);
        const int a_rd = LDS_A + arow * 128 + ((((arow >> 1) & 7) ^ (2 * h)) << 4);
        const int q = (lane >> 1) & 7;
        const int w_rd = LDS_W + (32 * h + q) * 256 + 8 * (lane & 1) + (((4 * wn + ((lane >> 4) & 1)) ^ (2 * q)) << 4);
        const unsigned pa0 = lds0 + a_rd, pa1 = lds0 + (a_rd ^ 16), pa2 = lds0 + (a_rd ^ 64), pa3 = lds0 + (a_rd ^ 80);
        const unsigned pb0 = lds0 + w_rd, pb1 = lds0 + (w_rd ^ 32), pb2 = pb0 + 2 * W_STAGE, pb3 = pb1 + 2 * W_STAGE;
        const unsigned done_addr = lane == 0 ? ctrl + (C_DONE + wave) * 4 : lds0 + LDS_DUMMY + lane * 4;
        const unsigned pmask = local ? 0u : 3u << (C_PROD + 2 * wm);
        unsigned cw, sn, st;
        __builtin_amdgcn_s_setprio(PRIO_M);
        asm volatile(
            ".macro AG_RDA t, lo, hi, off\n"
            "ds_read_b128 v[\\t:\\t+3], \\lo offset:\\off\n"
            "ds_read_b128 v[\\t+4:\\t+7], \\hi offset:\\off\n"
            ".endm\n"
            ".macro AG_RDB t, p, off\n"
            "ds_read_b64_tr_b8 v[\\t:\\t+1], \\p offset:\\off\n"
            "ds_read_b64_tr_b8 v[\\t+2:\\t+3], \\p offset:\\off+2048\n"
            "ds_read_b64_tr_b8 v[\\t+4:\\t+5], \\p offset:\\off+4096\n"
            "ds_read_b64_tr_b8 v[\\t+6:\\t+7], \\p offset:\\off+6144\n"
            ".endm\n"
            ".macro AG_MM c, a, b\n"
            "v_mfma_f32_32x32x64_f8f6f4 \\c, v[\\a:\\a+7], v[\\b:\\b+7], \\c cbsz:1 blgp:1\n"
            ".endm\n"
            ".macro AG_POLL mask\n"
            ".LAG_P\\@:\n"
            "ds_read_b32 %4, %15\n"
            "s_waitcnt lgkmcnt(0)\n"
            "v_cmp_lt_i32 vcc, %5, %4\n"
            "s_and_b32 %6, \\mask, vcc_lo\n"
            "s_cmp_eq_u32 %6, \\mask\n"
            "s_cbranch_scc1 .LAG_Q\\@\n"
            AG_CNT("s91")
            "s_sleep " SLEEP_M "\n"
            "s_branch .LAG_P\\@\n"
            ".LAG_Q\\@:\n"
            ".endm\n"
            ".macro AG_FETCH a0, a1, b0, b1, ao, q0, q1, wo\n"
            "AG_RDA \\a0, %7, %8, \\ao\n"
            "AG_RDB \\b0, \\q0, \\wo\n"
            "AG_RDA \\a1, %7, %8, \\ao+4096\n"
            "AG_RDB \\b1, \\q1, \\wo\n"
            ".endm\n"
            ".macro AG_HA a0, a1, b0, b1, f0, f1, ao, q0, q1, wo, look\n"
            "s_waitcnt lgkmcnt(6)\n"
            "AG_MM %0, \\a0, \\b0\n"
            "AG_RDA \\f0, %9, %10, \\ao\n"
            "AG_RDB \\f1, \\q0, \\wo+16384\n"
            "s_waitcnt lgkmcnt(10)\n"
            "AG_MM %1, \\a1, \\b0\n"
            "AG_RDA \\b0, %9, %10, \\ao+4096\n"
            ".if \\look\n"
            "ds_read_b32 %4, %15\n"
            ".endif\n"
            "s_waitcnt lgkmcnt(8+\\look)\n"
            "AG_MM %2, \\a0, \\b1\n"
            "AG_RDB \\a0, \\q1, \\wo+16384\n"
            "ds_add_u32 %16, %17\n"
            "AG_MM %3, \\a1, \\b1\n"
            "s_waitcnt lgkmcnt(5)\n"
            ".endm\n"
            ".macro AG_FOUR a0, a1, b0, b1\n"
            "AG_MM %0, \\a0, \\b0\n"
            "AG_MM %1, \\a1, \\b0\n"
            "s_waitcnt lgkmcnt(0)\n"
            "AG_MM %2, \\a0, \\b1\n"
            "AG_MM %3, \\a1, \\b1\n"
            ".endm\n"
            ".macro AG_STEP a0, a1, b0, b1, f0, f1, ao, q0, q1, wo, nao, nq0, nq1, nwo, mask\n"
            "s_add_u32 %5, %5, 1\n"
            "AG_HA \\a0, \\a1, \\b0, \\b1, \\f0, \\f1, \\ao, \\q0, \\q1, \\wo, 1\n"
            "v_cmp_lt_i32 vcc, %5, %4\n"
            "s_setprio 0\n"
            "s_and_b32 %6, %21, vcc_lo\n"
            "s_cbranch_scc0 .LAG_F\\@\n"
            "s_setprio 1\n"
            ".LAG_F\\@:\n"
            "s_and_b32 %6, \\mask, vcc_lo\n"
            "s_cmp_eq_u32 %6, \\mask\n"
            "s_cbranch_scc1 .LAG_R\\@\n"
            AG_CNT("s90")
            "AG_FOUR \\f0, \\b0, \\f1, \\a0\n"
            "AG_POLL \\mask\n"
            "AG_FETCH \\b1, \\f1, \\a1, \\f0, \\nao, \\nq0, \\nq1, \\nwo\n"
            "s_branch .LAG_E\\@\n"
            ".LAG_R\\@:\n"
            "AG_MM %0, \\f0, \\f1\n"
            "AG_RDA \\b1, %7, %8, \\nao\n"
            "AG_RDB \\a1, \\nq0, \\nwo\n"
            "AG_MM %1, \\b0, \\f1\n"
            "AG_RDA \\f1, %7, %8, \\nao+4096\n"
            "s_waitcnt lgkmcnt(9)\n"
            "AG_MM %2, \\f0, \\a0\n"
            "AG_RDB \\f0, \\nq1, \\nwo\n"
            "AG_MM %3, \\b0, \\a0\n"
            ".LAG_E\\@:\n"
            ".endm\n"
            "s_mov_b32 %5, 0\n"
            "AG_POLL %18\n"
            "AG_FETCH 80, 88, 96, 104, 0, %11, %12, 0\n"
            ".LAG_LOOP:\n"
            "AG_STEP 80, 88, 96, 104, 112, 120, 0, %11, %12, 0, 16384, %11, %12, 32768, %19\n"
            "s_cmp_eq_u32 %5, 31\n"
            "s_cbranch_scc1 .LAG_LAST\n"
            "AG_STEP 104, 120, 88, 112, 80, 96, 16384, %11, %12, 32768, 32768, %13, %14, 0, %20\n"
            "AG_STEP 112, 96, 120, 80, 104, 88, 32768, %13, %14, 0, 0, %11, %12, 0, %18\n"
            "s_branch .LAG_LOOP\n"
            ".LAG_LAST:\n"
            "AG_HA 104, 120, 88, 112, 80, 96, 16384, %11, %12, 32768, 0\n"
            "AG_FOUR 80, 88, 96, 104\n"
            "s_nop 15\n"
            "s_nop 3\n"
            : "+v"(acc[0][0]), "+v"(acc[1][0]), "+v"(acc[0][1]), "+v"(acc[1][1]), "=&v"(cw), "=&s"(sn), "=&s"(st)
            : "v"(pa0), "v"(pa1), "v"(pa2), "v"(pa3), "v"(pb0), "v"(pb1), "v"(pb2), "v"(pb3),
              "v"(ctrl_lane), "v"(done_addr), "v"(1u),
              "s"(pmask | (1u << C_WFULL)), "s"(pmask | (2u << C_WFULL)), "s"(pmask | (4u << C_WFULL)),
              "s"(1u << (C_DONE + (wave ^ 4)))
            : "memory", "vcc", "scc",
              "v80", "v81", "v82", "v83", "v84", "v85", "v86", "v87", "v88", "v89", "v90", "v91", "v92", "v93", "v94", "v95",
              "v96", "v97", "v98", "v99", "v100", "v101", "v102", "v103", "v104", "v105", "v106", "v107", "v108", "v109",
              "v110", "v111", "v112", "v113", "v114", "v115", "v116", "v117", "v118", "v119", "v120", "v121", "v122",
              "v123", "v124", "v125", "v126", "v127");
        static_assert(A_STAGE == 16384 && W_STAGE == 32768 && NSTEP == 32, "offsets in the K loop");
        const unsigned bad = __builtin_amdgcn_readlane(cw, C_BAD);
        STAMP_T(0, 9);
        STAMP_T(448, 13);

        if (local) ctrl_wait<8>(ctrl_lane, 1u << C_SENT, LW - 1);
        if (bad) sc = __builtin_nanf("");
        const int m0 = (int)src * MLOC + rb * BM + wm * 64;
        const __amdgpu_buffer_rsrc_t ro = __builtin_amdgcn_make_buffer_rsrc(
            outp + (size_t)m0 * NLOC + n0 + wn * 64, 0, 64 * NLOC * 4, 0x00020000);
        const int vo = (4 * (lane >> 5)) * (NLOC * 4) + (lane & 31) * 4;
#define AG_STORE(AUX) \
        _Pragma("unroll") for (int i = 0; i < 2; ++i) \
        _Pragma("unroll") for (int r = 0; r < 16; ++r) \
        _Pragma("unroll") for (int j = 0; j < 2; ++j) \
            __builtin_amdgcn_raw_buffer_store_b32(__builtin_bit_cast(unsigned, silu_out(acc[i][j][r], sc)), ro, \
                                                  vo + j * 128, (32 * i + (r & 3) + 8 * (r >> 2)) * (NLOC * 4), AUX)
        if (OUT_AUX0 == OUT_AUX1 || local || ((SKEW >> (4 * rb)) & 15) == 0) { AG_STORE(OUT_AUX0); }
        else { AG_STORE(OUT_AUX1); }
    } else if (wave < 13 - LW && local) {
        __builtin_amdgcn_s_barrier();
    } else if (wave < 12 && !local) {
        const int pw = wave - 8;
        unsigned* cntp = (unsigned*)(myws + WS_CNT) + 8 * b + pw;
        const unsigned cnt = __hip_atomic_load(cntp, __ATOMIC_RELAXED, __HIP_MEMORY_SCOPE_AGENT);
        __builtin_amdgcn_s_barrier();
        __builtin_amdgcn_s_setprio(PRIO_X);
        const unsigned call = __builtin_amdgcn_readfirstlane(cnt) + 1u;
        if (lane == 0) __hip_atomic_store(cntp, call, __ATOMIC_RELAXED, __HIP_MEMORY_SCOPE_AGENT);
        const unsigned par = call & 1u;
        const int grp = 4 * rb + pw;
        const unsigned info_addr = lds0 + LDS_INFO + (lane & 31) * 16 + pw * 4;
        const unsigned long long lb = (unsigned long long)(myws + src * SRC_BYTES + par * LAND_BYTES + grp * PIECE);
        const u32x4 land = {(unsigned)lb, (unsigned)(lb >> 32) & 0xFFFFu, (unsigned)(LAND_BYTES - grp * PIECE), 0x00020000u};
        const unsigned prod_addr = ctrl + (C_PROD + pw) * 4;
        const unsigned sink = lds0 + LDS_DUMMY + lane * 4;
        unsigned ok, t0, t1, t2, t3, t4, t5, t6, t7;
        asm volatile(
            ".macro XP_LOOK\n"
            "ds_read_b32 v112, %11\n"
            "s_waitcnt lgkmcnt(0)\n"
            "v_cmp_ne_u32 vcc, 0, v112\n"
            "s_mov_b32 %2, vcc_lo\n"
            "s_bitcmp1_b32 %2, %3\n"
            ".endm\n"
            ".macro XP_POLL\n"
            ".LXP_P\\@:\n"
            "XP_LOOK\n"
            "s_cbranch_scc1 .LXP_Q\\@\n"
            "s_sleep 1\n"
            "s_branch .LXP_P\\@\n"
            ".LXP_Q\\@:\n"
            ".endm\n"
            ".macro XP_ISSUE t, inf\n"
            "v_readlane_b32 \\inf, v112, %3\n"
            "s_lshl_b32 %7, %3, 16\n"
            "s_bitcmp1_b32 \\inf, 9\n"
            "s_cbranch_scc1 .LXP_BAIL\n"
            "buffer_load_dwordx4 v[\\t:\\t+3], %9, %18, %7 offen sc0 sc1\n"
            "s_bitcmp1_b32 \\inf, 7\n"
            "s_cselect_b64 vcc, -1, 0\n"
            "buffer_load_dwordx4 v[\\t+4:\\t+7], %9, %18, %7 offen offset:1024 sc0 sc1\n"
            "v_cndmask_b32 v113, %10, %9, vcc\n"
            "buffer_load_dwordx4 v[\\t+8:\\t+11], %9, %18, %7 offen offset:2048 sc0 sc1\n"
            "buffer_load_dwordx4 v[\\t+12:\\t+15], v113, %18, %7 offen offset:3072 sc0 sc1\n"
            ".endm\n"
            ".macro XP_FREE\n"
            "s_sub_u32 %7, %1, 2\n"
            ".LXP_F\\@:\n"
            "s_waitcnt lgkmcnt(0)\n"
            "v_cmp_lt_i32 vcc, %7, v114\n"
            "s_and_b32 %8, %19, vcc_lo\n"
            "s_cmp_eq_u32 %8, %19\n"
            "s_cbranch_scc1 .LXP_G\\@\n"
            "s_sleep 1\n"
            "ds_read_b32 v114, %12\n"
            "s_branch .LXP_F\\@\n"
            ".LXP_G\\@:\n"
            ".endm\n"
            ".macro XP_XAD t, r\n"
            "v_xad_u32 v[\\t], v[\\t], v[\\r], %7\n"
            "v_xad_u32 v[\\t+1], v[\\t+1], v[\\r+1], %7\n"
            "v_xad_u32 v[\\t+2], v[\\t+2], v[\\r+2], %7\n"
            "v_xad_u32 v[\\t+3], v[\\t+3], v[\\r+3], %7\n"
            ".endm\n"
            ".macro XP_TOP d, i\n"
            "v_lshrrev_b32 v[104+\\i], 5, v[104+\\i]\n"
            "v_lshrrev_b32 v[96+\\i], 5, v[96+\\i]\n"
            "v_lshrrev_b32 v[100+\\i], 3, v[100+\\i]\n"
            "v_perm_b32 v[104+\\i], %8, %8, v[104+\\i]\n"
            "v_add3_u32 v[\\d], v[96+\\i], v[100+\\i], v[104+\\i]\n"
            ".endm\n"
            ".macro XP_DECODE t, inf\n"
            "v_add_u32 v115, %6, %16\n"
            "s_bitcmp1_b32 \\inf, 7\n"
            "s_cbranch_scc0 .LXP_D\\@\n"
            "XP_FREE\n"
            "ds_write_b128 v115, v[\\t:\\t+3]\n"
            "ds_write_b128 v115, v[\\t+4:\\t+7] offset:1024\n"
            "ds_write_b128 v115, v[\\t+8:\\t+11] offset:2048\n"
            "ds_write_b128 v115, v[\\t+12:\\t+15] offset:3072\n"
            "s_branch .LXP_E\\@\n"
            ".LXP_D\\@:\n"
            ".irp i, 0, 1, 2, 3, 4, 5, 6, 7, 8, 9, 10, 11\n"
            "v_and_b32 v[96+\\i], %17, v[\\t+\\i]\n"
            ".endr\n"
            "v_lshrrev_b32 v116, %15, v[\\t+12]\n"
            "XP_FREE\n"
            "s_and_b32 %7, \\inf, 0x7f\n"
            "s_mul_i32 %7, %7, 0x01010101\n"
            "s_add_u32 %8, %7, 0x90801000\n"
            "XP_XAD \\t, 96\n"
            "ds_write_b128 v115, v[\\t:\\t+3]\n"
            "XP_XAD \\t+4, 100\n"
            "ds_write_b128 v115, v[\\t+4:\\t+7] offset:1024\n"
            "XP_XAD \\t+8, 104\n"
            "ds_write_b128 v115, v[\\t+8:\\t+11] offset:2048\n"
            "XP_TOP \\t+12, 0\n"
            "XP_TOP \\t+13, 1\n"
            "XP_TOP \\t+14, 2\n"
            "XP_TOP \\t+15, 3\n"
            "ds_write_b128 v115, v[\\t+12:\\t+15] offset:3072\n"
            "s_add_u32 %7, %6, %20\n"
            "v_and_b32 v117, 3, v116\n"
            "v_lshlrev_b32 v118, 1, v116\n"
            "v_cmp_eq_u32 vcc, 0, v117\n"
            "v_and_or_b32 v118, v118, 24, 5\n"
            "v_bfe_u32 v116, v116, 4, 10\n"
            "v_lshlrev_b32 v117, v118, v117\n"
            "v_lshl_add_u32 v116, v116, 2, %7\n"
            "v_cndmask_b32 v116, v116, %14, vcc\n"
            "ds_xor_b32 v116, v117\n"
            ".LXP_E\\@:\n"
            "ds_add_u32 %13, v119\n"
            "s_add_u32 %6, %6, 16384\n"
            "s_cmp_eq_u32 %6, 49152\n"
            "s_cselect_b32 %6, 0, %6\n"
            ".endm\n"
            ".macro XP_BODY cur, cinf, nxt, ninf\n"
            "s_bitcmp1_b32 %2, %3\n"
            "s_cbranch_scc1 .LXP_I\\@\n"
            "XP_LOOK\n"
            "s_cbranch_scc1 .LXP_I\\@\n"
            "s_waitcnt vmcnt(0)\n"
            "XP_LOOK\n"
            "s_cbranch_scc0 .LXP_J\\@\n"
            ".LXP_I\\@:\n"
            "XP_ISSUE \\nxt, \\ninf\n"
            ".LXP_J\\@:\n"
            "ds_read_b32 v114, %12\n"
            "s_waitcnt vmcnt(4)\n"
            "XP_DECODE \\cur, \\cinf\n"
            "s_bitcmp1_b32 %2, %3\n"
            "s_cbranch_scc1 .LXP_K\\@\n"
            "XP_POLL\n"
            "XP_ISSUE \\nxt, \\ninf\n"
            ".LXP_K\\@:\n"
            "s_mov_b32 %1, %3\n"
            "s_add_u32 %3, %3, 1\n"
            ".endm\n"
            "s_mov_b32 %0, 0\n"
            "s_mov_b32 %1, 0\n"
            "s_mov_b32 %3, 0\n"
            "s_mov_b32 %6, 0\n"
            "v_mov_b32 v119, 1\n"
            "XP_POLL\n"
            "XP_ISSUE 64, %4\n"
            "s_mov_b32 %3, 1\n"
            ".LXP_LOOP:\n"
            "XP_BODY 64, %4, 80, %5\n"
            "s_cmp_eq_u32 %1, 31\n"
            "s_cbranch_scc1 .LXP_LAST\n"
            "XP_BODY 80, %5, 64, %4\n"
            "s_branch .LXP_LOOP\n"
            ".LXP_LAST:\n"
            "ds_read_b32 v114, %12\n"
            "s_waitcnt vmcnt(0)\n"
            "XP_DECODE 80, %5\n"
            "s_mov_b32 %0, 1\n"
            ".LXP_BAIL:\n"
            "s_waitcnt vmcnt(0) lgkmcnt(0)\n"
            : "=&s"(ok), "=&s"(t0), "=&s"(t1), "=&s"(t2), "=&s"(t3), "=&s"(t4), "=&s"(t5), "=&s"(t6), "=&s"(t7)
            : "v"(lane * 16), "v"((lane >> 1) * 4), "v"(info_addr), "v"(ctrl_lane), "v"(lane == 0 ? prod_addr : sink), "v"(sink),
              "v"(16 * (lane & 1)), "v"(lds0 + LDS_A + pw * PIECE + lane * 16), "v"(0x60606060u),
              "s"(land), "s"(0xFu << (C_DONE + 4 * (pw >> 1))), "s"(lds0 + LDS_A + pw * PIECE)
            : "memory", "vcc", "scc",
              "v64", "v65", "v66", "v67", "v68", "v69", "v70", "v71", "v72", "v73", "v74", "v75", "v76", "v77", "v78", "v79",
              "v80", "v81", "v82", "v83", "v84", "v85", "v86", "v87", "v88", "v89", "v90", "v91", "v92", "v93", "v94", "v95",
              "v96", "v97", "v98", "v99", "v100", "v101", "v102", "v103", "v104", "v105", "v106", "v107",
              "v112", "v113", "v114", "v115", "v116", "v117", "v118", "v119");
        static_assert(NGRP * PIECE == 65536 && INFO_RAW == 0x80u && INFO_SKIP == 0x200u, "constants of the x loop");
        if (!ok) ctrl_set(prod_addr, (unsigned)NSTEP);
        STAMP_T(512, 8);
        STAMP_T(704, 14);
    } else if (wave == 14 && !local) {
        unsigned* cntp = (unsigned*)(myws + WS_CNT) + 8 * b + 5;
        const unsigned cnt = __hip_atomic_load(cntp, __ATOMIC_RELAXED, __HIP_MEMORY_SCOPE_AGENT);
        __builtin_amdgcn_s_barrier();
        __builtin_amdgcn_s_setprio(PRIO_X);
        const unsigned call = __builtin_amdgcn_readfirstlane(cnt) + 1u;
        if (lane == 0) __hip_atomic_store(cntp, call, __ATOMIC_RELAXED, __HIP_MEMORY_SCOPE_AGENT);
        const unsigned long long* fl = (const unsigned long long*)(myws + src * SRC_BYTES + SRC_FLAG) + ((call & 1u) * 4 + rb) * (4 * NSTEP) + lane;
        const unsigned long long t0 = __builtin_amdgcn_s_memrealtime();
        for (int half = 0; half < 2; ++half) {
            unsigned long long seen0 = 0, seen1 = 0;
            for (;;) {
                const unsigned long long v0 = __hip_atomic_load(fl + 64 * half, __ATOMIC_RELAXED, __HIP_MEMORY_SCOPE_SYSTEM);
                unsigned long long v1 = 0;
                const bool both = half == 0 && (seen0 >> 40) != 0;
                if (both) v1 = __hip_atomic_load(fl + 64, __ATOMIC_RELAXED, __HIP_MEMORY_SCOPE_SYSTEM);
                const unsigned tm = __hip_atomic_load(tmo, __ATOMIC_RELAXED, __HIP_MEMORY_SCOPE_AGENT);
                if (__builtin_amdgcn_readfirstlane(tm) == call || __builtin_amdgcn_s_memrealtime() - t0 > DEADLINE_TICKS) {
                    if (lane == 0) __hip_atomic_store(tmo, call, __ATOMIC_RELAXED, __HIP_MEMORY_SCOPE_AGENT);
                    ctrl_set(ctrl + C_BAD * 4, 1u);
                    ctrl_set(lds0 + LDS_INFO + lane * 4, INFO_SKIP);
                    ctrl_set(lds0 + LDS_INFO + 256 + lane * 4, INFO_SKIP);
                    half = 2;
                    break;
                }
                const bool up0 = (unsigned)(v0 >> 32) >= call, up1 = (unsigned)(v1 >> 32) >= call;
                const unsigned long long m0 = __builtin_amdgcn_ballot_w64(up0), m1 = __builtin_amdgcn_ballot_w64(up1);
                if (m0 != seen0) ctrl_set(lds0 + LDS_INFO + half * 256 + lane * 4, up0 ? (unsigned)v0 | INFO_UP : 0u);
                if (m1 != seen1) ctrl_set(lds0 + LDS_INFO + 256 + lane * 4, up1 ? (unsigned)v1 | INFO_UP : 0u);
                seen0 = m0; seen1 = m1;
                if (m0 == ~0ull) break;
                __builtin_amdgcn_s_sleep(SLEEP_P);
            }
        }
    } else if (wave >= 12 && wave < 15) {
        const int ld = wave - 12, nld = local ? 3 : 2;
        PROF_DECL;
        const int wr = lane >> 4;
        const unsigned wo0 = (unsigned)(wr * NLOC + n0 + (((lane & 15) ^ (2 * wr)) << 4));
        const unsigned wo1 = (unsigned)((wr + 4) * NLOC + n0 + (((lane & 15) ^ (2 * (wr + 4))) << 4)) - 1024u;
        const unsigned wo2 = wo0 + 8 * NLOC - 2048u;
        const unsigned wo3 = wo1 + 8 * NLOC - 2048u;
        const unsigned xo0 = (lane >> 3) * KDIM + (((lane & 7) ^ (lane >> 4)) << 4);
        const unsigned xo1 = (xo0 ^ 64u) + 8 * KDIM - 1024, xo2 = xo0 + 16 * KDIM - 2048, xo3 = (xo0 ^ 64u) + 24 * KDIM - 3072;
        __builtin_amdgcn_s_barrier();
        __builtin_amdgcn_s_setprio(PRIO_W);
        for (int i = 0; i < ld; ++i) __builtin_amdgcn_s_sleep(16);
        unsigned hold;
        for (int s = ld; s < NSTEP; s += nld) {
            const int sl = s % 3;
            ctrl_wait<SLEEP_H>(ctrl_lane, 0xFFu << C_DONE, s - 2);
            PROF(s < 6 ? 3 : 0);
            const unsigned char* g = p.w + (size_t)s * (BK * NLOC);
            const unsigned l = lds0 + LDS_W + sl * W_STAGE;
            auto stand_back = [&]() {
                while (__builtin_amdgcn_readfirstlane(hold) != 0u) { __builtin_amdgcn_s_sleep(1); hold = ctrl_get(ctrl + C_HOLD * 4); }
                hold = *(volatile LAS unsigned*)(size_t)(ctrl + C_HOLD * 4);
            };
            hold = *(volatile LAS unsigned*)(size_t)(ctrl + C_HOLD * 4);
#pragma unroll
            for (int i = 0; i < 8; ++i) {
                stand_back();
                dma4(g + i * 16 * NLOC, wo0, wo1, wo2, wo3, l + i * 4096);
            }
            if (local) {
                const unsigned char* gx = p.x + (size_t)(rb * BM) * KDIM + s * BK;
#pragma unroll
                for (int i = 0; i < 4; ++i) {
                    stand_back();
                    dma4(gx + i * 32 * KDIM, xo0, xo1, xo2, xo3, lds0 + LDS_A + sl * A_STAGE + i * PIECE);
                }
            }
            PROF(s < 6 ? 3 : 1);
            asm volatile("s_waitcnt vmcnt(0)" ::: "memory");
            PROF(s < 6 ? 3 : 2);
            ctrl_set1(ctrl + (C_WFULL + sl) * 4, (unsigned)(s + 1));
        }
        PROF_OUT(768, 8);
    } else {
        __builtin_amdgcn_s_setprio(PRIO_S);
        const int sw = wave == 15 ? 4 : wave - 8;
        int ord = local ? ord0 + 32 * (sw - (5 - LW)) : ord0;
        const int lst_off = LDS_LIST + sw * 256;
        const int ca = (lane >> 3) * KDIM + (((lane & 7) ^ (lane >> 4)) << 4), cb = ca ^ 64;
        u32x4 x0, x1, x2, x3;
        int piece;
        auto load_x = [&]() {
            piece = piece_at(ord, lane);
            const unsigned char* xg = p.x + (size_t)(piece & 15) * (32 * KDIM) + (piece >> 4) * BK;
            x0 = *(const u32x4*)(xg + ca);
            x1 = *(const u32x4*)(xg + 8 * KDIM + cb);
            x2 = *(const u32x4*)(xg + 16 * KDIM + ca);
            x3 = *(const u32x4*)(xg + 24 * KDIM + cb);
        };
        load_x();
        unsigned* cntp = (unsigned*)(myws + WS_CNT) + 8 * b + sw;
        const unsigned cnt = __hip_atomic_load(cntp, __ATOMIC_RELAXED, __HIP_MEMORY_SCOPE_AGENT);
        char* wsl = p.ws[lane >> 3];
        char* wsd[NR];
#pragma unroll
        for (int j = 1; j < NR; ++j) { wsd[j] = p.ws[j]; asm volatile("" : "+s"(wsd[j])); }
        asm volatile("" ::: "memory");
        __builtin_amdgcn_s_barrier();
        const unsigned long long tstart = __builtin_amdgcn_s_memrealtime();
        STAMP_T(960, 2);
        const unsigned call = __builtin_amdgcn_readfirstlane(cnt) + 1u;
        const unsigned par = call & 1u;
        const unsigned o = lane * 16;
        unsigned* tok = (unsigned*)(myws + WS_TOK);
        unsigned long long* late = (unsigned long long*)(myws + WS_LATE);
        for (;;) {
            const int step = piece >> 4, grp = piece & 15;
            bool raw = true;
            int lo = 0;
            u32x4 d0 = x0, d1 = x1, d2 = x2, lst = x3;
            if (ord >= RAW_FIRST) {
                if (lane < 33) ctrl_set(lds0 + lst_off + lane * 4, 0u);
                unsigned sum = 0;
#pragma unroll
                for (int d = 0; d < 4; ++d) sum = __builtin_amdgcn_sad_u8(x0[d] & 0x7F7F7F7Fu, 0u, sum);
                sum += __builtin_amdgcn_update_dpp(0u, sum, 0x111, 0xf, 0xf, false);
                sum += __builtin_amdgcn_update_dpp(0u, sum, 0x112, 0xf, 0xf, false);
                sum += __builtin_amdgcn_update_dpp(0u, sum, 0x114, 0xf, 0xf, false);
                sum += __builtin_amdgcn_update_dpp(0u, sum, 0x118, 0xf, 0xf, false);
                const int tot = __builtin_amdgcn_readlane(sum, 15) + __builtin_amdgcn_readlane(sum, 31) +
                                __builtin_amdgcn_readlane(sum, 47) + __builtin_amdgcn_readlane(sum, 63);
                lo = (tot - 20582) >> 10;
                lo = lo < 0 ? 0 : (lo > 96 ? 96 : lo);
                const unsigned lo4 = (unsigned)lo * 0x01010101u;
                unsigned m[4];
                u32x4 c0, c1, c2;
#pragma unroll
                for (int i = 0; i < 4; ++i) {
                    const u32x3 c = pack((u32x4){x0[i], x1[i], x2[i], x3[i]}, lo4, m[i]);
                    c0[i] = c[0]; c1[i] = c[1]; c2[i] = c[2];
                }
                unsigned n = 0;
#pragma unroll
                for (int v = 0; v < 4; ++v) n += __builtin_popcount((m[v] | (m[v] >> 1)) & 0x55555555u);
                if (n) {
                    unsigned slot;
                    asm volatile("ds_add_rtn_u32 %0, %1, %2\n\ts_waitcnt lgkmcnt(0)" : "=&v"(slot) : "v"(lds0 + lst_off + 128), "v"(n) : "memory");
#pragma unroll
                    for (int v = 0; v < 4; ++v) {
                        unsigned mv = m[v];
                        while (mv != 0 && slot < LIST_CAP) {
                            const unsigned bit = (unsigned)__builtin_ctz(mv) & ~1u;
                            const unsigned dl = (mv >> bit) & 3u;
                            mv &= ~(3u << bit);
                            const unsigned pos = ((bit >> 1) & 3) * 1024 + lane * 16 + v * 4 + (bit >> 3);
                            *(LAS unsigned short*)(smem + lst_off + slot * 2) = (unsigned short)((pos << 2) | dl);
                            ++slot;
                        }
                    }
                }
                asm volatile("" ::: "memory");
                lst = *(volatile LAS u32x4*)(smem + lst_off + (lane & 7) * 16);
                raw = __builtin_amdgcn_readfirstlane(ctrl_get(lds0 + lst_off + 128)) > LIST_CAP;
                if (!raw) { d0 = c0; d1 = c1; d2 = c2; }
            }
            const size_t off = par * LAND_BYTES + (size_t)piece * PIECE;
            unsigned long long* fp = (unsigned long long*)(wsl + SRC_FLAG) + ((par * 4 + (grp >> 2)) * NSTEP + step) * 4 + (grp & 3);
            char* lp = wsl + off + 3072 + (lane & 7) * 16;
            asm volatile("" : "+v"(d0), "+v"(d1), "+v"(d2), "+v"(lp));
            if (ord < 32) STAMP_T(960, 12);
            unsigned long long due = tstart + (unsigned long long)(OFFSET + (ord * PERIOD10 + (ord < RAW_FIRST ? ord : RAW_FIRST) * 16) / 10);
            if (ord >= SEND_WIN) {
                const unsigned long long due0 = due;
                for (;;) {
                    while (__builtin_amdgcn_s_memrealtime() + (local ? 160 : 70) < due) __builtin_amdgcn_s_sleep(2);
                    const unsigned tk = __hip_atomic_load(tok + (ord - SEND_WIN - (local ? 16 : 0)) * 16, __ATOMIC_RELAXED, __HIP_MEMORY_SCOPE_AGENT);
                    const unsigned long long lt = __hip_atomic_load(late, __ATOMIC_RELAXED, __HIP_MEMORY_SCOPE_AGENT);
                    if (__builtin_amdgcn_readfirstlane((unsigned)(lt >> 32)) == call) due = due0 + __builtin_amdgcn_readfirstlane((unsigned)lt);
                    const unsigned long long now = __builtin_amdgcn_s_memrealtime();
                    if (now > due0 + 20000) break;
                    if (__builtin_amdgcn_readfirstlane(tk) >= call) {
                        if (now + (local ? 160 : 70) >= due) break;
                    } else if (now > due && lane == 0) {
                        __hip_atomic_fetch_max(late, ((unsigned long long)call << 32) | (unsigned)(now - due0), __ATOMIC_RELAXED, __HIP_MEMORY_SCOPE_AGENT);
                    }
                }
            } else {
                while (__builtin_amdgcn_s_memrealtime() + 30 < due) __builtin_amdgcn_s_sleep(2);
            }
            while (__builtin_amdgcn_s_memrealtime() + HOLD_LEAD < due) __builtin_amdgcn_s_sleep(1);
            if (lane == 0) asm volatile("ds_add_u32 %0, %1" :: "v"(ctrl + C_HOLD * 4), "v"(1u) : "memory");
            while (__builtin_amdgcn_s_memrealtime() < due) __builtin_amdgcn_s_sleep(1);
#pragma unroll
            for (int j = 1; j < NR; ++j) {
                st3k(wsd[j] + off, o, d0, d1, d2);
                if (raw) st1k(wsd[j] + off, o, x3);
            }
            if (!raw && lane >= 8) st_ptr(lp, lst);
            if (lane == 0) asm volatile("ds_sub_u32 %0, %1" :: "v"(ctrl + C_HOLD * 4), "v"(1u) : "memory");
            asm volatile("s_waitcnt vmcnt(0)" ::: "memory");
            if (lane >= 8 && (lane & 7) == 0)
                __hip_atomic_store(fp, ((unsigned long long)call << 32) | (raw ? INFO_RAW : (unsigned)lo),
                                   __ATOMIC_RELAXED, __HIP_MEMORY_SCOPE_SYSTEM);
            if (lane == 0 && local) asm volatile("ds_add_u32 %0, %1" :: "v"(ctrl + C_SENT * 4), "v"(1u) : "memory");
            if (lane == 0) __hip_atomic_store(tok + ord * 16, call, __ATOMIC_RELAXED, __HIP_MEMORY_SCOPE_AGENT);
            ord += 224;
            if (ord >= ord_end) break;
            load_x();
        }
        if (lane == 0) __hip_atomic_store(cntp, call, __ATOMIC_RELAXED, __HIP_MEMORY_SCOPE_AGENT);
    }
}

}

extern "C" void kernel_launch(int rank, int nranks, ncclComm_t comm,
                              hipStream_t stream,
                              void* const* d_in, const int* in_sizes, int n_in,
                              void* d_out, int out_size,
                              const int* in_dtypes, int out_dtype,
                              void* const* d_out_all,
                              void* const* d_ws_all, size_t ws_bytes) {
    (void)comm; (void)in_dtypes; (void)out_dtype; (void)d_out_all;
    if (nranks != NR || (unsigned)rank >= (unsigned)NR || n_in != 4 || ws_bytes < WS_NEED ||
        in_sizes[0] != MLOC * KDIM || in_sizes[1] != KDIM * NLOC || out_size != NR * MLOC * NLOC) {
        fprintf(stderr, "ag_gemm: unexpected problem shape\n");
        return;
    }
    HIP_CHECK(hipFuncSetAttribute((const void*)ag_gemm, hipFuncAttributeMaxDynamicSharedMemorySize, LDS_BYTES));
    Params p;
    p.x = (const unsigned char*)d_in[0];
    p.w = (const unsigned char*)d_in[1];
    p.sx = (const float*)d_in[2];
    p.sw = (const float*)d_in[3];
    p.out = (float*)d_out;
    for (int j = 0; j < NR; ++j) p.ws[j] = (char*)d_ws_all[(rank + j) % NR] + (j ? rank * SRC_BYTES : 0);
    p.rank = rank;
    p.pad = 0;
    hipLaunchKernelGGL(ag_gemm, dim3(GRID), dim3(NTHREADS), LDS_BYTES, stream, p);
    HIP_CHECK(hipGetLastError());
}
```

```cpp
#include <hip/hip_runtime.h>
#include <rccl/rccl.h>
#include <cstdio>

#define HIP_CHECK(expr) do { hipError_t _e = (expr); if (_e != hipSuccess) { \
    fprintf(stderr, "HIP error %s at %s:%d\n", hipGetErrorString(_e), __FILE__, __LINE__); \
    return; } } while (0)

constexpr int PERIOD = 570;
constexpr int OFFSET = 30;
constexpr int SEND_WIN = 72;
constexpr int LW = 3;
constexpr int RAW_FIRST = 16;
constexpr int EARLY = 32;
constexpr int HOLD_LEAD = 50;
constexpr int X_NOM = 60;
constexpr int WARM0 = 40;
constexpr int WARMG = 20;
constexpr int AHEAD = 5;
constexpr int DMA_NOP = 5;
constexpr int DMA_NOP_L = 1;
constexpr int NTX = 4;
constexpr int PACK_AHEAD = 96;
constexpr int FASTW = 28;
constexpr int CAPT = 3420;
constexpr int CODE_BYTES = 16384;

namespace {

constexpr int NR = 8;
constexpr int KDIM = 4096;
constexpr int MLOC = 512;
constexpr int NLOC = 2048;
constexpr int BM = 128, BN = 256, BK = 128;
constexpr int NSTEP = KDIM / BK;
constexpr int NGRP = MLOC / 32;
constexpr int NPIECE = NSTEP * NGRP;
constexpr int GRID = 256;
constexpr int NTHREADS = 1024;

constexpr int A_STAGE = BM * BK;
constexpr int W_STAGE = BN * BK;
constexpr int LDS_A = 0;
constexpr int LDS_W = 3 * A_STAGE;
constexpr int LDS_CTRL = LDS_W + 3 * W_STAGE;
constexpr int LDS_LIST = LDS_CTRL + 128;
constexpr int LDS_DUMMY = LDS_LIST + 2 * 256;
constexpr int LDS_INFO = LDS_DUMMY + 256;
constexpr int LDS_STASH = LDS_INFO + 512;
constexpr int LDS_BYTES = LDS_STASH + 2 * 4096;
constexpr int C_PROD = 0;
constexpr int C_WFULL = 4;
constexpr int C_BAD = 7;
constexpr int C_HOLD = 16;
constexpr int C_DONE = 8;

constexpr int LIST_CAP = 64;
constexpr int PIECE = 4096;
constexpr unsigned INFO_RAW = 0x80u, INFO_UP = 0x100u, INFO_SKIP = 0x200u;

constexpr size_t LAND_BYTES = (size_t)NPIECE * PIECE;
constexpr size_t SRC_FLAG = 2 * LAND_BYTES, SRC_BYTES = SRC_FLAG + 65536;
constexpr size_t WS_CNT = NR * SRC_BYTES;
constexpr size_t WS_TOK = WS_CNT + GRID * 64;
constexpr size_t WS_LATE = WS_TOK + NPIECE * 64;
constexpr size_t WS_TMO = WS_LATE + 256;
constexpr size_t WS_NEED = WS_TMO + 256;

constexpr unsigned long long DEADLINE_TICKS = 100000000ull;

typedef unsigned u32x3 __attribute__((ext_vector_type(3)));
typedef unsigned u32x4 __attribute__((ext_vector_type(4)));
#define LAS __attribute__((address_space(3)))
#define GAS __attribute__((address_space(1)))

struct Params {
    const float* sx;
    const float* sw;
    float* out;
    int rank;
    int pad;
    const unsigned char* x;
    const unsigned char* w;
    char* ws[NR];
};

__host__ __device__ __forceinline__ int row_block(int b) { return ((b & 7) / (8 / NTX)) * (32 / NTX) + (b >> 3) / NTX; }
__host__ __device__ __forceinline__ int col_tile(int b) { return ((b & 7) % (8 / NTX)) * NTX + (b >> 3) % NTX; }
__host__ __device__ __forceinline__ int src_offset(int b) { return row_block(b) & (NR - 1); }

template <int PAD>
__device__ __forceinline__ void dma4(const void* base, unsigned v0, unsigned v1, unsigned v2, unsigned v3, unsigned lds) {
    unsigned keep;
    asm volatile("s_mov_b32 %0, m0\n\ts_mov_b32 m0, %6\n\ts_nop 2\n\t"
                 "global_load_lds_dwordx4 %1, %5\n\ts_nop %7\n\t"
                 "global_load_lds_dwordx4 %2, %5 offset:1024\n\ts_nop %7\n\t"
                 "global_load_lds_dwordx4 %3, %5 offset:2048\n\ts_nop %7\n\t"
                 "global_load_lds_dwordx4 %4, %5 offset:3072\n\t"
                 "s_mov_b32 m0, %0"
                 : "=&s"(keep) : "v"(v0), "v"(v1), "v"(v2), "v"(v3), "s"(base), "s"(lds), "n"(PAD) : "memory");
}

__device__ __forceinline__ void ctrl_set(unsigned addr, unsigned val) {
    asm volatile("ds_write_b32 %0, %1" :: "v"(addr), "v"(val) : "memory");
}
__device__ __forceinline__ void ctrl_set1(unsigned addr, unsigned val) {
    asm volatile("s_mov_b64 exec, 1\n\tds_write_b32 %0, %1\n\ts_mov_b64 exec, -1" :: "v"(addr), "v"(val) : "memory");
}
__device__ __forceinline__ unsigned ctrl_get(unsigned addr) {
    unsigned v;
    asm volatile("ds_read_b32 %0, %1\n\ts_waitcnt lgkmcnt(0)" : "=v"(v) : "v"(addr) : "memory");
    return v;
}

__device__ __forceinline__ u32x3 pack(u32x4 x, unsigned lo4, unsigned& m) {
    unsigned f[4], z[4];
#pragma unroll
    for (int d = 0; d < 4; ++d) {
        const unsigned y = ((x[d] | 0x80808080u) - lo4) & 0x1F1F1F1Fu;
        z[d] = ((y + lo4) ^ x[d]) & 0x60606060u;
        f[d] = y | (x[d] & 0x80808080u);
    }
    m = (z[0] >> 5) | (z[1] >> 3) | (z[2] >> 1) | (z[3] << 1);
    u32x3 c;
    c[0] = f[0] | ((f[3] & 0x03030303u) << 5);
    c[1] = f[1] | ((f[3] & 0x0C0C0C0Cu) << 3);
    c[2] = f[2] | ((f[3] & 0x10101010u) << 1) | ((f[3] & 0x80808080u) >> 1);
    return c;
}

template <int OFF>
__device__ __forceinline__ void st7(const unsigned long long (&base)[NR], unsigned o, u32x4 a) {
    asm volatile("s_nop 4\n\t"
                 "global_store_dwordx4 %0, %1, %2 offset:%9 sc0 sc1\n\t"
                 "global_store_dwordx4 %0, %1, %3 offset:%9 sc0 sc1\n\t"
                 "global_store_dwordx4 %0, %1, %4 offset:%9 sc0 sc1\n\t"
                 "global_store_dwordx4 %0, %1, %5 offset:%9 sc0 sc1\n\t"
                 "global_store_dwordx4 %0, %1, %6 offset:%9 sc0 sc1\n\t"
                 "global_store_dwordx4 %0, %1, %7 offset:%9 sc0 sc1\n\t"
                 "global_store_dwordx4 %0, %1, %8 offset:%9 sc0 sc1\n\ts_nop 1"
                 :: "v"(o), "v"(a), "s"(base[1]), "s"(base[2]), "s"(base[3]), "s"(base[4]), "s"(base[5]), "s"(base[6]),
                    "s"(base[7]), "n"(OFF) : "memory");
}
__device__ __forceinline__ void st_ptr(char* p, u32x4 a) {
    asm volatile("global_store_dwordx4 %0, %1, off sc0 sc1\n\ts_nop 1" :: "v"(p), "v"(a) : "memory");
}

__global__ void __launch_bounds__(NTHREADS, 4) ag_gemm(Params p) {
    extern __shared__ __attribute__((aligned(16))) char smem_[];
    LAS char* smem = (LAS char*)smem_;
    const unsigned lds0 = (unsigned)(size_t)smem;
    const int b = blockIdx.x & (GRID - 1);
    const int tid = (threadIdx.x + NTHREADS - 64) & (NTHREADS - 1);
    const int lane = tid & 63;
    const int wave = __builtin_amdgcn_readfirstlane(tid >> 6);
    unsigned long long code0;
    asm volatile("s_getpc_b64 %0" : "=s"(code0));
    unsigned long long peer[NR];
#pragma unroll
    for (int j = 1; j < NR; ++j) peer[j] = (unsigned long long)p.ws[j];
    asm volatile("" : "+s"(peer[1]), "+s"(peer[2]), "+s"(peer[3]), "+s"(peer[4]), "+s"(peer[5]), "+s"(peer[6]), "+s"(peer[7])
                 : "s"(p.x), "s"(p.w), "s"(p.ws[0]), "s"(p.rank), "s"(p.sx), "s"(p.sw), "s"(p.out));
    auto peer_of_lane = [&]() {
        unsigned long long v = peer[1];
#pragma unroll
        for (int j = 2; j < NR; ++j) { v = (lane >> 3) == j ? peer[j] : v; asm volatile("" : "+v"(v)); }
        return v;
    };
    char* myws = p.ws[0];

    const unsigned src = (unsigned)(p.rank + src_offset(b)) & (NR - 1);
    const bool local = src_offset(b) == 0;
    const int rb = row_block(b) >> 3, nt = col_tile(b);
    const int n0 = nt * BN;
    static_assert(NTX <= 4, "four local tiles per XCD");
    const int lq = (b >> 3) / (8 * NTX);
    const int rr = ((b >> 3) - NTX * (1 + lq)) * 8 + (b & 7);
    const int ord0 = local ? NPIECE - 32 * LW + (lq * NTX + (b >> 3) % NTX) * 8 + (b & 7) : rr;
    const int ord_end = local ? NPIECE : NPIECE - 32 * LW;
    const unsigned ctrl = lds0 + LDS_CTRL;
    const unsigned ctrl_lane = ctrl + (lane & 15) * 4;
    unsigned* tmo = (unsigned*)(myws + WS_TMO);

    const bool sender = wave == 15 || (local && wave == 14);
    int ord = local ? ord0 + 32 * (15 - wave) : ord0;
    const int ca = (lane >> 3) * KDIM + (((lane & 7) ^ (lane >> 4)) << 4), cb = ca ^ 64;
    u32x4 x0, x1, x2, x3;
    int piece;
    auto load_x = [&]() {
        piece = (ord & ~15) | ((ord + 4 * p.rank) & 15);
        const unsigned char* xg = p.x + (size_t)(piece & 15) * (32 * KDIM) + (piece >> 4) * BK;
        x0 = *(const u32x4*)(xg + ca);
        x1 = *(const u32x4*)(xg + 8 * KDIM + cb);
        x2 = *(const u32x4*)(xg + 16 * KDIM + ca);
        x3 = *(const u32x4*)(xg + 24 * KDIM + cb);
    };
    const int ord_step = local ? 64 : 224;
    int ahead = ord >= PACK_AHEAD && ord + ord_step < ord_end;
    if (ahead) ord += ord_step;
    if (sender && ord < NPIECE) load_x();
    unsigned* cntp = (unsigned*)(myws + WS_CNT) + 16 * b + wave;
    unsigned cnt = 0;
    if (wave >= 8) cnt = __hip_atomic_load(cntp, __ATOMIC_RELAXED, __HIP_MEMORY_SCOPE_AGENT);
    if (wave == 0) {
        ctrl_set(ctrl + (lane & 31) * 4, (lane & 24) == C_DONE || ((lane & 31) == C_HOLD && ord0 < EARLY) ? 1u : 0u);
        ctrl_set(lds0 + LDS_INFO + lane * 4, local ? INFO_UP | INFO_RAW : 0u);
        ctrl_set(lds0 + LDS_INFO + 256 + lane * 4, local ? INFO_UP | INFO_RAW : 0u);
        asm volatile("s_waitcnt lgkmcnt(0)" ::: "memory");
    }
    asm volatile("" ::: "memory");
    __builtin_amdgcn_s_barrier();
    asm volatile("" ::: "memory");
    const unsigned long long tstart = __builtin_amdgcn_s_memrealtime();

    if (wave < 8) {
        if (wave >= 2 && wave < 6 && (b >> 3) == 13) {
            const unsigned long long c = code0 + ((wave - 2) * 64 + lane) * 128;
            if (((wave - 2) * 64 + lane) * 128 < CODE_BYTES)
                __builtin_amdgcn_global_load_lds((const GAS void*)c, (LAS void*)(smem + LDS_DUMMY), 4, 0, 0);
        }
        if (wave == 1 && (b >> 3) == 12 && lane >= 8 && (lane & 7) < 3) {
            const unsigned long long t = peer_of_lane() + (lane & 7) * LAND_BYTES + (b & 7) * 128;
            __builtin_amdgcn_global_load_lds((const GAS void*)t, (LAS void*)(smem + LDS_DUMMY), 4, 0, 17);
        }
        if (wave == 1 && (b >> 3) == 14 && lane < 40) {
            const char* t = lane < 16 ? (const char*)p.out + (size_t)lane * (2u << 20)
                          : lane < 36 ? myws + (size_t)min(lane - 16, 16) * (2u << 20) : (const char*)p.w + (size_t)(lane - 36) * (2u << 20);
            __builtin_amdgcn_global_load_lds((const GAS void*)(t + (b & 7) * 128), (LAS void*)(smem + LDS_DUMMY), 4, 0, 17);
        }
        float sc = __builtin_bit_cast(float, __builtin_amdgcn_readfirstlane(__builtin_bit_cast(unsigned, p.sx[0] * p.sw[0])));
        float* outp = p.out;
        asm volatile("" : "+s"(sc), "+s"(outp));

        if (!local && rr >= 64) {
            while (__builtin_amdgcn_s_memrealtime() < tstart + WARM0 + WARMG * wave) __builtin_amdgcn_s_sleep(1);
            constexpr int XL = MLOC * KDIM / 128, WL = KDIM * NLOC / 128, W0 = 3 * BK * NLOC / 128;
            const int li = (wave * 160 + rr - 64) * 64 + lane;
            const unsigned char* q = li < XL ? p.x + (size_t)li * 128 : p.w + (size_t)(li - XL + W0) * 128;
            if (li < XL + WL - W0) __builtin_amdgcn_global_load_lds((const GAS void*)q, (LAS void*)(smem + LDS_DUMMY), 4, 0, 0);
        }
        const int wm = wave >> 2, wn = wave & 3;
        const int h = lane >> 5;
        const int arow = 64 * wm + (lane & 31);
        const int a_rd = LDS_A + arow * 128 + ((((arow >> 1) & 7) ^ (2 * h)) << 4);
        const int q = (lane >> 1) & 7;
        const int w_rd = LDS_W + (32 * h + q) * 256 + 8 * (lane & 1) + (((4 * wn + ((lane >> 4) & 1)) ^ (2 * q)) << 4);
        const unsigned pa0 = lds0 + a_rd, pa1 = lds0 + (a_rd ^ 16), pa2 = lds0 + (a_rd ^ 64), pa3 = lds0 + (a_rd ^ 80);
        const unsigned pb0 = lds0 + w_rd, pb1 = lds0 + (w_rd ^ 32), pb2 = pb0 + 2 * W_STAGE, pb3 = pb1 + 2 * W_STAGE;
        const unsigned done_addr = lane == 0 ? ctrl + (C_DONE + wave) * 4 : lds0 + LDS_DUMMY + lane * 4;
        const unsigned pmask = 3u << (C_PROD + 2 * wm);
        unsigned cw, sn, st;
        const float* ob = outp + (size_t)((int)src * MLOC + rb * BM + wm * 64) * NLOC + n0 + wn * 64;
        const int vo = (4 * (lane >> 5)) * (NLOC * 4) + (lane & 31) * 4;
        asm volatile(
            ".macro AG_RDA t, lo, hi, off\n"
            "ds_read_b128 v[\\t:\\t+3], \\lo offset:\\off\n"
            "ds_read_b128 v[\\t+4:\\t+7], \\hi offset:\\off\n"
            ".endm\n"
            ".macro AG_RDB t, p, off\n"
            "ds_read_b64_tr_b8 v[\\t:\\t+1], \\p offset:\\off\n"
            "ds_read_b64_tr_b8 v[\\t+2:\\t+3], \\p offset:\\off+2048\n"
            "ds_read_b64_tr_b8 v[\\t+4:\\t+5], \\p offset:\\off+4096\n"
            "ds_read_b64_tr_b8 v[\\t+6:\\t+7], \\p offset:\\off+6144\n"
            ".endm\n"
            ".macro AG_MM c, a, b\n"
            "v_mfma_f32_32x32x64_f8f6f4 \\c, v[\\a:\\a+7], v[\\b:\\b+7], \\c cbsz:1 blgp:1\n"
            ".endm\n"
            ".macro AG_POLL mask\n"
            ".LAG_P\\@:\n"
            "ds_read_b32 %[cw], %[cl]\n"
            "s_waitcnt lgkmcnt(0)\n"
            "v_cmp_lt_i32 vcc, %[sn], %[cw]\n"
            "s_and_b32 %[st], \\mask, vcc_lo\n"
            "s_cmp_eq_u32 %[st], \\mask\n"
            "s_cbranch_scc1 .LAG_Q\\@\n"
            "s_sleep 1\n"
            "s_branch .LAG_P\\@\n"
            ".LAG_Q\\@:\n"
            ".endm\n"
            ".macro AG_FETCH a0, a1, b0, b1, ao, q0, q1, wo\n"
            "AG_RDA \\a0, %[pa0], %[pa1], \\ao\n"
            "AG_RDB \\b0, \\q0, \\wo\n"
            "AG_RDA \\a1, %[pa0], %[pa1], \\ao+4096\n"
            "AG_RDB \\b1, \\q1, \\wo\n"
            ".endm\n"
            ".macro AG_HA a0, a1, b0, b1, f0, f1, ao, q0, q1, wo\n"
            "s_waitcnt lgkmcnt(6)\n"
            "AG_MM v[16:31], \\a0, \\b0\n"
            "s_waitcnt lgkmcnt(4)\n"
            "AG_MM v[32:47], \\a1, \\b0\n"
            "s_waitcnt lgkmcnt(0)\n"
            "AG_MM v[48:63], \\a0, \\b1\n"
            "AG_RDA \\f0, %[pa2], %[pa3], \\ao\n"
            "AG_RDB \\f1, \\q0, \\wo+16384\n"
            "AG_RDA \\b0, %[pa2], %[pa3], \\ao+4096\n"
            "AG_RDB \\a0, \\q1, \\wo+16384\n"
            "ds_add_u32 %[da], %[one]\n"
            "AG_MM v[64:79], \\a1, \\b1\n"
            "s_waitcnt lgkmcnt(7)\n"
            ".endm\n"
            ".macro AG_FOUR a0, a1, b0, b1\n"
            "AG_MM v[16:31], \\a0, \\b0\n"
            "s_waitcnt lgkmcnt(5)\n"
            "AG_MM v[32:47], \\a1, \\b0\n"
            "s_waitcnt lgkmcnt(0)\n"
            "AG_MM v[48:63], \\a0, \\b1\n"
            "AG_MM v[64:79], \\a1, \\b1\n"
            ".endm\n"
            ".macro AG_STEP a0, a1, b0, b1, f0, f1, ao, q0, q1, wo, nao, nq0, nq1, nwo, mask\n"
            "s_add_u32 %[sn], %[sn], 1\n"
            "AG_HA \\a0, \\a1, \\b0, \\b1, \\f0, \\f1, \\ao, \\q0, \\q1, \\wo\n"
            "AG_MM v[16:31], \\f0, \\f1\n"
            "ds_read_b32 %[cw], %[cl]\n"
            "s_waitcnt lgkmcnt(6)\n"
            "AG_MM v[32:47], \\b0, \\f1\n"
            "s_waitcnt lgkmcnt(2)\n"
            "AG_MM v[48:63], \\f0, \\a0\n"
            "s_waitcnt lgkmcnt(0)\n"
            "v_cmp_lt_i32 vcc, %[sn], %[cw]\n"
            "s_and_b32 %[st], \\mask, vcc_lo\n"
            "s_cmp_eq_u32 %[st], \\mask\n"
            "s_cbranch_scc1 .LAG_R\\@\n"
            "AG_MM v[64:79], \\b0, \\a0\n"
            "AG_POLL \\mask\n"
            "AG_FETCH \\b1, \\f1, \\a1, \\f0, \\nao, \\nq0, \\nq1, \\nwo\n"
            "s_branch .LAG_E\\@\n"
            ".LAG_R\\@:\n"
            "AG_FETCH \\b1, \\f1, \\a1, \\f0, \\nao, \\nq0, \\nq1, \\nwo\n"
            "AG_MM v[64:79], \\b0, \\a0\n"
            ".LAG_E\\@:\n"
            "s_setprio 0\n"
            "s_and_b32 %[st], %[pm], vcc_lo\n"
            "s_cbranch_scc0 .LAG_F\\@\n"
            "s_setprio 1\n"
            ".LAG_F\\@:\n"
            ".endm\n"
            ".macro AG_ST p, k\n"
            ".if \\p\n"
            ".if (\\k & 1)\n"
            "buffer_store_dword v[\\p+32+(\\k>>1)], %[vo], s[84:87], %[st] offen offset:128 sc1\n"
            "s_add_u32 %[st], %[st], 8192+(\\k/7)*4*8192\n"
            ".else\n"
            "buffer_store_dword v[\\p+(\\k>>1)], %[vo], s[84:87], %[st] offen sc1\n"
            ".endif\n"
            ".endif\n"
            ".endm\n"
            ".macro AG_YIELD\n"
            "s_cmp_lg_u32 %[so], 0\n"
            "s_cbranch_scc1 .LAG_Z\\@\n"
            ".LAG_Y\\@:\n"
            "ds_read_b32 v104, %[cl] offset:64\n"
            "s_waitcnt lgkmcnt(0)\n"
            "v_readfirstlane_b32 s82, v104\n"
            "s_cmp_eq_u32 s82, 0\n"
            "s_cbranch_scc1 .LAG_Z\\@\n"
            "s_sleep 2\n"
            "s_branch .LAG_Y\\@\n"
            ".LAG_Z\\@:\n"
            ".endm\n"
            ".macro AG_EPI c, p\n"
            "AG_YIELD\n"
            "s_cbranch_vccnz .LAG_C\\@\n"
            "s_waitcnt vmcnt(2)\n"
            ".LAG_C\\@:\n"
            "AG_ST \\p, 0\n"
            ".irp k, 0, 2, 32, 34\n"
            "v_pk_mul_f32 v[96+(\\k&2)+(\\k>>3):97+(\\k&2)+(\\k>>3)], v[\\c+\\k:\\c+\\k+1], v[122:123]\n"
            ".endr\n"
            ".irp k, 96, 97, 98\n"
            "v_med3_f32 v[\\k], v[\\k], -v126, v126\n"
            ".endr\n"
            "AG_ST \\p, 1\n"
            ".irp k, 99, 100, 101, 102, 103\n"
            "v_med3_f32 v[\\k], v[\\k], -v126, v126\n"
            ".endr\n"
            "v_exp_f32 v96, v96\n"
            "AG_ST \\p, 2\n"
            ".irp k, 97, 98, 99, 100\n"
            "v_exp_f32 v[\\k], v[\\k]\n"
            ".endr\n"
            "AG_ST \\p, 3\n"
            ".irp k, 101, 102, 103\n"
            "v_exp_f32 v[\\k], v[\\k]\n"
            ".endr\n"
            "v_pk_mul_f32 v[\\c:\\c+1], v[\\c:\\c+1], v[120:121]\n"
            "AG_ST \\p, 4\n"
            ".irp k, 2, 32, 34\n"
            "v_pk_mul_f32 v[\\c+\\k:\\c+\\k+1], v[\\c+\\k:\\c+\\k+1], v[120:121]\n"
            ".endr\n"
            ".irp k, 96, 98, 100\n"
            "v_pk_add_f32 v[\\k:\\k+1], v[\\k:\\k+1], v[124:125]\n"
            ".endr\n"
            "AG_ST \\p, 5\n"
            "v_pk_add_f32 v[102:103], v[102:103], v[124:125]\n"
            ".irp k, 96, 97, 98\n"
            "v_rcp_f32 v[\\k], v[\\k]\n"
            ".endr\n"
            "AG_ST \\p, 6\n"
            ".irp k, 99, 100, 101, 102\n"
            "v_rcp_f32 v[\\k], v[\\k]\n"
            ".endr\n"
            "AG_ST \\p, 7\n"
            "v_rcp_f32 v103, v103\n"
            ".irp k, 0, 32, 2, 34\n"
            "v_pk_mul_f32 v[\\c+\\k:\\c+\\k+1], v[\\c+\\k:\\c+\\k+1], v[96+(\\k&2)+(\\k>>3):97+(\\k&2)+(\\k>>3)]\n"
            ".endr\n"
            ".endm\n"
            "s_mov_b64 s[84:85], %[ob]\n"
            "s_mov_b32 s86, 64*8192\n"
            "s_mov_b32 s87, 0x00020000\n"
            "s_mov_b32 s88, %[sc]\n"
            ".irp k, 16, 18, 20, 22, 24, 26, 28, 30, 32, 34, 36, 38, 40, 42, 44, 46, 48, 50, 52, 54, 56, 58, 60, 62, 64, 66, 68, 70, 72, 74, 76, 78\n"
            "v_mov_b64 v[\\k:\\k+1], 0\n"
            ".endr\n"
            "s_mov_b32 %[sn], 0\n"
            "AG_POLL %[m0]\n"
            "AG_FETCH 80, 88, 96, 104, 0, %[pb0], %[pb1], 0\n"
            ".LAG_LOOP:\n"
            "AG_STEP 80, 88, 96, 104, 112, 120, 0, %[pb0], %[pb1], 0, 16384, %[pb0], %[pb1], 32768, %[m1]\n"
            "s_cmp_eq_u32 %[sn], 31\n"
            "s_cbranch_scc1 .LAG_LAST\n"
            "AG_STEP 104, 120, 88, 112, 80, 96, 16384, %[pb0], %[pb1], 32768, 32768, %[pb2], %[pb3], 0, %[m2]\n"
            "AG_STEP 112, 96, 120, 80, 104, 88, 32768, %[pb2], %[pb3], 0, 0, %[pb0], %[pb1], 0, %[m0]\n"
            "s_branch .LAG_LOOP\n"
            ".LAG_LAST:\n"
            "AG_HA 104, 120, 88, 112, 80, 96, 16384, %[pb0], %[pb1], 32768\n"
            "AG_FOUR 80, 88, 96, 104\n"
            "v_readlane_b32 %[st], %[cw], 7\n"
            "s_cmp_lg_u32 %[st], 0\n"
            "s_cselect_b32 s88, 0x7fc00000, s88\n"
            "s_setprio 0\n"
            "s_memrealtime s[82:83]\n"
            "s_waitcnt lgkmcnt(0)\n"
            "s_sub_u32 s82, s82, %[tc]\n"
            "s_cmp_gt_i32 s82, 0\n"
            "s_cselect_b64 vcc, -1, 0\n"
            "v_mov_b32 v120, s88\n"
            "v_mov_b32 v121, s88\n"
            "v_mul_f32 v122, 0xbfb8aa3b, v120\n"
            "v_mul_f32 v123, 0xbfb8aa3b, v120\n"
            "v_mov_b32 v124, 1.0\n"
            "v_mov_b32 v125, 1.0\n"
            "v_mov_b32 v126, 0x42ad1f97\n"
            "s_mov_b32 %[st], 0\n"
            "s_nop 7\n"
            "AG_EPI 16, 0\n"
            "AG_EPI 20, 16\n"
            "AG_EPI 24, 20\n"
            "AG_EPI 28, 24\n"
            "AG_EPI 32, 28\n"
            "AG_EPI 36, 32\n"
            "AG_EPI 40, 36\n"
            "AG_EPI 44, 40\n"
            ".irp k, 0, 1, 2, 3, 4, 5, 6, 7\n"
            "AG_ST 44, \\k\n"
            ".endr\n"
            : [cw] "=&v"(cw), [sn] "=&s"(sn), [st] "=&s"(st)
            : [pa0] "v"(pa0), [pa1] "v"(pa1), [pa2] "v"(pa2), [pa3] "v"(pa3), [pb0] "v"(pb0), [pb1] "v"(pb1), [pb2] "v"(pb2),
              [pb3] "v"(pb3), [cl] "v"(ctrl_lane), [da] "v"(done_addr), [one] "v"(1u), [vo] "v"(vo),
              [m0] "s"(pmask | (1u << C_WFULL)), [m1] "s"(pmask | (2u << C_WFULL)), [m2] "s"(pmask | (4u << C_WFULL)),
              [pm] "s"(1u << (C_DONE + (wave ^ 4))), [ob] "s"(ob), [sc] "s"(sc), [so] "s"(src_offset(b)), [tc] "s"((unsigned)tstart + CAPT)
            : "memory", "vcc", "scc", "s82", "s83", "s84", "s85", "s86", "s87", "s88",
              "v16", "v17", "v18", "v19", "v20", "v21", "v22", "v23", "v24", "v25", "v26", "v27", "v28", "v29", "v30",
              "v31", "v32", "v33", "v34", "v35", "v36", "v37", "v38", "v39", "v40", "v41", "v42", "v43", "v44", "v45",
              "v46", "v47", "v48", "v49", "v50", "v51", "v52", "v53", "v54", "v55", "v56", "v57", "v58", "v59", "v60",
              "v61", "v62", "v63", "v64", "v65", "v66", "v67", "v68", "v69", "v70", "v71", "v72", "v73", "v74", "v75",
              "v76", "v77", "v78", "v79", "v80", "v81", "v82", "v83", "v84", "v85", "v86", "v87", "v88", "v89", "v90",
              "v91", "v92", "v93", "v94", "v95", "v96", "v97", "v98", "v99", "v100", "v101", "v102", "v103", "v104", "v105",
              "v106", "v107", "v108", "v109", "v110", "v111", "v112", "v113", "v114", "v115", "v116", "v117", "v118",
              "v119", "v120", "v121", "v122", "v123", "v124", "v125", "v126", "v127");
        static_assert(A_STAGE == 16384 && W_STAGE == 32768 && NSTEP == 32 && NLOC * 4 == 8192 && C_BAD == 7 && C_HOLD == 16, "constants of the K loop");
    } else if (wave < 12) {
        const int pw = wave - 8;
        __builtin_amdgcn_s_setprio(2);
        const unsigned call = __builtin_amdgcn_readfirstlane(cnt) + 1u;
        if (lane == 0) __hip_atomic_store(cntp, call, __ATOMIC_RELAXED, __HIP_MEMORY_SCOPE_AGENT);
        const unsigned par = call & 1u;
        const int grp = 4 * rb + pw;
        const unsigned info_addr = lds0 + LDS_INFO + (lane & 31) * 16 + pw * 4;
        const unsigned long long lb = local ? (unsigned long long)(p.x + (size_t)grp * 32 * KDIM)
                                            : (unsigned long long)(myws + src * SRC_BYTES + par * LAND_BYTES + grp * PIECE);
        const u32x4 land = {(unsigned)lb, (unsigned)(lb >> 32) & 0xFFFFu, local ? 32u * KDIM : (unsigned)(LAND_BYTES - grp * PIECE), 0x00020000u};
        const unsigned xo0 = local ? ca : lane * 16, xo1 = local ? cb + 8 * KDIM : lane * 16 + 1024;
        const unsigned xo2 = local ? ca + 16 * KDIM : lane * 16 + 2048, xo3 = local ? cb + 24 * KDIM : lane * 16 + 3072;
        const unsigned prod_addr = ctrl + (C_PROD + pw) * 4;
        const unsigned sink = lds0 + LDS_DUMMY + lane * 4;
        unsigned ok, t0, t1, t2, t3, t4, t5, t6, t7;
        asm volatile(
            ".macro XP_LOOK\n"
            "ds_read_b32 v112, %11\n"
            "s_waitcnt lgkmcnt(0)\n"
            "v_cmp_ne_u32 vcc, 0, v112\n"
            "s_mov_b32 %2, vcc_lo\n"
            "s_bitcmp1_b32 %2, %3\n"
            ".endm\n"
            ".macro XP_POLL\n"
            ".LXP_P\\@:\n"
            "XP_LOOK\n"
            "s_cbranch_scc1 .LXP_Q\\@\n"
            "s_sleep 1\n"
            "s_branch .LXP_P\\@\n"
            ".LXP_Q\\@:\n"
            ".endm\n"
            ".macro XP_ISSUE t, inf\n"
            "v_readlane_b32 \\inf, v112, %3\n"
            "s_lshl_b32 %7, %3, %24\n"
            "s_bitcmp1_b32 \\inf, 9\n"
            "s_cbranch_scc1 .LXP_BAIL\n"
            "buffer_load_dwordx4 v[\\t:\\t+3], %9, %18, %7 offen sc0 sc1\n"
            "buffer_load_dwordx4 v[\\t+4:\\t+7], %21, %18, %7 offen sc0 sc1\n"
            "buffer_load_dwordx4 v[\\t+8:\\t+11], %22, %18, %7 offen sc0 sc1\n"
            "s_bitcmp1_b32 \\inf, 7\n"
            "s_cbranch_scc0 .LXP_L\\@\n"
            "buffer_load_dwordx4 v[\\t+12:\\t+15], %23, %18, %7 offen sc0 sc1\n"
            "s_branch .LXP_M\\@\n"
            ".LXP_L\\@:\n"
            "buffer_load_dword v[\\t+12], %10, %18, %7 offen offset:3072 sc0 sc1\n"
            ".LXP_M\\@:\n"
            ".endm\n"
            ".macro XP_FREE\n"
            "s_sub_u32 %7, %1, 2\n"
            ".LXP_F\\@:\n"
            "s_waitcnt lgkmcnt(0)\n"
            "v_cmp_lt_i32 vcc, %7, v114\n"
            "s_and_b32 %8, %19, vcc_lo\n"
            "s_cmp_eq_u32 %8, %19\n"
            "s_cbranch_scc1 .LXP_G\\@\n"
            "s_sleep 1\n"
            "ds_read_b32 v114, %12\n"
            "s_branch .LXP_F\\@\n"
            ".LXP_G\\@:\n"
            ".endm\n"
            ".macro XP_XAD t, r\n"
            "v_xad_u32 v[\\t], v[\\t], v[\\r], %7\n"
            "v_xad_u32 v[\\t+1], v[\\t+1], v[\\r+1], %7\n"
            "v_xad_u32 v[\\t+2], v[\\t+2], v[\\r+2], %7\n"
            "v_xad_u32 v[\\t+3], v[\\t+3], v[\\r+3], %7\n"
            ".endm\n"
            ".macro XP_TOP d, i\n"
            "v_lshrrev_b32 v[104+\\i], 5, v[104+\\i]\n"
            "v_lshrrev_b32 v[96+\\i], 5, v[96+\\i]\n"
            "v_lshrrev_b32 v[100+\\i], 3, v[100+\\i]\n"
            "v_perm_b32 v[104+\\i], %8, %8, v[104+\\i]\n"
            "v_add3_u32 v[\\d], v[96+\\i], v[100+\\i], v[104+\\i]\n"
            ".endm\n"
            ".macro XP_DECODE t, inf\n"
            "v_add_u32 v115, %6, %16\n"
            "s_bitcmp1_b32 \\inf, 7\n"
            "s_cbranch_scc0 .LXP_D\\@\n"
            "XP_FREE\n"
            "s_setprio 1\n"
            "ds_write_b128 v115, v[\\t:\\t+3]\n"
            "ds_write_b128 v115, v[\\t+4:\\t+7] offset:1024\n"
            "ds_write_b128 v115, v[\\t+8:\\t+11] offset:2048\n"
            "ds_write_b128 v115, v[\\t+12:\\t+15] offset:3072\n"
            "s_branch .LXP_E\\@\n"
            ".LXP_D\\@:\n"
            "s_setprio 0\n"
            ".irp i, 0, 1, 2, 3, 4, 5, 6, 7, 8, 9, 10, 11\n"
            "v_and_b32 v[96+\\i], %17, v[\\t+\\i]\n"
            ".endr\n"
            "v_lshrrev_b32 v116, %15, v[\\t+12]\n"
            "XP_FREE\n"
            "s_and_b32 %7, \\inf, 0x7f\n"
            "s_mul_i32 %7, %7, 0x01010101\n"
            "s_add_u32 %8, %7, 0x90801000\n"
            "XP_XAD \\t, 96\n"
            "s_setprio 1\n"
            "ds_write_b128 v115, v[\\t:\\t+3]\n"
            "s_setprio 0\n"
            "XP_XAD \\t+4, 100\n"
            "s_setprio 1\n"
            "ds_write_b128 v115, v[\\t+4:\\t+7] offset:1024\n"
            "s_setprio 0\n"
            "XP_XAD \\t+8, 104\n"
            "s_setprio 1\n"
            "ds_write_b128 v115, v[\\t+8:\\t+11] offset:2048\n"
            "s_setprio 0\n"
            "XP_TOP \\t+12, 0\n"
            "XP_TOP \\t+13, 1\n"
            "XP_TOP \\t+14, 2\n"
            "XP_TOP \\t+15, 3\n"
            "s_add_u32 %7, %6, %20\n"
            "v_and_b32 v117, 3, v116\n"
            "v_lshlrev_b32 v118, 1, v116\n"
            "v_cmp_eq_u32 vcc, 0, v117\n"
            "v_and_or_b32 v118, v118, 24, 5\n"
            "v_bfe_u32 v116, v116, 4, 10\n"
            "v_lshlrev_b32 v117, v118, v117\n"
            "v_lshl_add_u32 v116, v116, 2, %7\n"
            "v_cndmask_b32 v116, v116, %14, vcc\n"
            "s_setprio 1\n"
            "ds_write_b128 v115, v[\\t+12:\\t+15] offset:3072\n"
            "ds_xor_b32 v116, v117\n"
            ".LXP_E\\@:\n"
            "ds_add_u32 %13, v119\n"
            "s_setprio 2\n"
            "s_add_u32 %6, %6, 16384\n"
            "s_cmp_eq_u32 %6, 49152\n"
            "s_cselect_b32 %6, 0, %6\n"
            ".endm\n"
            ".macro XP_BODY cur, cinf, nxt, ninf\n"
            "s_bitcmp1_b32 %2, %3\n"
            "s_cbranch_scc1 .LXP_I\\@\n"
            "XP_LOOK\n"
            "s_cbranch_scc1 .LXP_I\\@\n"
            "s_waitcnt vmcnt(0)\n"
            "XP_LOOK\n"
            "s_cbranch_scc0 .LXP_J\\@\n"
            ".LXP_I\\@:\n"
            "XP_ISSUE \\nxt, \\ninf\n"
            ".LXP_J\\@:\n"
            "ds_read_b32 v114, %12\n"
            "s_waitcnt vmcnt(4)\n"
            "XP_DECODE \\cur, \\cinf\n"
            "s_bitcmp1_b32 %2, %3\n"
            "s_cbranch_scc1 .LXP_K\\@\n"
            "XP_POLL\n"
            "XP_ISSUE \\nxt, \\ninf\n"
            ".LXP_K\\@:\n"
            "s_mov_b32 %1, %3\n"
            "s_add_u32 %3, %3, 1\n"
            ".endm\n"
            "s_mov_b32 %0, 0\n"
            "s_mov_b32 %1, 0\n"
            "s_mov_b32 %3, 0\n"
            "s_mov_b32 %6, 0\n"
            "v_mov_b32 v119, 1\n"
            "XP_POLL\n"
            "XP_ISSUE 64, %4\n"
            "s_mov_b32 %3, 1\n"
            ".LXP_LOOP:\n"
            "XP_BODY 64, %4, 80, %5\n"
            "s_cmp_eq_u32 %1, 31\n"
            "s_cbranch_scc1 .LXP_LAST\n"
            "XP_BODY 80, %5, 64, %4\n"
            "s_branch .LXP_LOOP\n"
            ".LXP_LAST:\n"
            "ds_read_b32 v114, %12\n"
            "s_waitcnt vmcnt(0)\n"
            "XP_DECODE 80, %5\n"
            "s_mov_b32 %0, 1\n"
            ".LXP_BAIL:\n"
            "s_waitcnt vmcnt(0) lgkmcnt(0)\n"
            : "=&s"(ok), "=&s"(t0), "=&s"(t1), "=&s"(t2), "=&s"(t3), "=&s"(t4), "=&s"(t5), "=&s"(t6), "=&s"(t7)
            : "v"(xo0), "v"((lane >> 1) * 4), "v"(info_addr), "v"(ctrl_lane), "v"(lane == 0 ? prod_addr : sink), "v"(sink),
              "v"(16 * (lane & 1)), "v"(lds0 + LDS_A + pw * PIECE + lane * 16), "v"(0x60606060u),
              "s"(land), "s"(0xFu << (C_DONE + 4 * (pw >> 1))), "s"(lds0 + LDS_A + pw * PIECE),
              "v"(xo1), "v"(xo2), "v"(xo3), "s"(local ? 7 : 16)
            : "memory", "vcc", "scc",
              "v64", "v65", "v66", "v67", "v68", "v69", "v70", "v71", "v72", "v73", "v74", "v75", "v76", "v77", "v78", "v79",
              "v80", "v81", "v82", "v83", "v84", "v85", "v86", "v87", "v88", "v89", "v90", "v91", "v92", "v93", "v94", "v95",
              "v96", "v97", "v98", "v99", "v100", "v101", "v102", "v103", "v104", "v105", "v106", "v107",
              "v112", "v113", "v114", "v115", "v116", "v117", "v118", "v119");
        static_assert(NGRP * PIECE == 65536 && INFO_RAW == 0x80u && INFO_SKIP == 0x200u, "constants of the x loop");
        if (!ok) ctrl_set(prod_addr, (unsigned)NSTEP);
    } else if (wave == 14 && !local) {
        __builtin_amdgcn_s_setprio(2);
        const unsigned call = __builtin_amdgcn_readfirstlane(cnt) + 1u;
        if (lane == 0) __hip_atomic_store(cntp, call, __ATOMIC_RELAXED, __HIP_MEMORY_SCOPE_AGENT);
        const unsigned long long* fl = (const unsigned long long*)(myws + src * SRC_BYTES + SRC_FLAG) + ((call & 1u) * 4 + rb) * (4 * NSTEP) + lane;
        const unsigned long long t0 = __builtin_amdgcn_s_memrealtime();
        int lim = AHEAD;
        int ws = 0;
        unsigned long long seen = 0;
        while (ws < NSTEP) {
            if (lim < NSTEP - 1) {
                const unsigned cw = ctrl_get(ctrl_lane);
                const unsigned p01 = min(__builtin_amdgcn_readlane(cw, C_PROD), __builtin_amdgcn_readlane(cw, C_PROD + 1));
                const unsigned p23 = min(__builtin_amdgcn_readlane(cw, C_PROD + 2), __builtin_amdgcn_readlane(cw, C_PROD + 3));
                lim = min((int)min(p01, p23) + AHEAD, NSTEP - 1);
                if (ws > lim - 1 && lim < NSTEP - 1) { __builtin_amdgcn_s_sleep(2); continue; }
            }
            if (ws >= FASTW) {
                const unsigned long long full = ~0ull >> (64 - 4 * (NSTEP - ws));
                unsigned long long m_, sv_;
                unsigned n_;
                asm volatile(
                    ".macro PL_LD r\n"
                    "global_load_dwordx2 v[\\r:\\r+1], %[fa], off sc0 sc1\n"
                    "global_load_dword v[\\r+2], %[ta], off sc1\n"
                    ".endm\n"
                    ".macro PL_SEE r\n"
                    "s_waitcnt vmcnt(4)\n"
                    "v_cmp_eq_u32 vcc, %[call], v[\\r+2]\n"
                    "s_cbranch_vccnz .LPL_END\n"
                    "v_cmp_le_u32 %[m], %[call], v[\\r+1]\n"
                    "v_or_b32 v[\\r], 0x100, v[\\r]\n"
                    "s_cmp_eq_u64 %[m], %[seen]\n"
                    "s_cbranch_scc1 .LPL_N\\@\n"
                    "s_mov_b64 exec, %[m]\n"
                    "ds_write_b32 %[ia], v[\\r]\n"
                    "s_mov_b64 exec, %[full]\n"
                    "s_mov_b64 %[seen], %[m]\n"
                    "s_cmp_eq_u64 %[m], %[full]\n"
                    "s_cbranch_scc1 .LPL_END\n"
                    ".LPL_N\\@:\n"
                    "s_sleep 6\n"
                    ".endm\n"
                    "s_mov_b64 %[sv], exec\n"
                    "s_mov_b64 exec, %[full]\n"
                    "s_mov_b32 %[n], 24\n"
                    "PL_LD 64\n"
                    "s_sleep 6\n"
                    "PL_LD 68\n"
                    "s_sleep 6\n"
                    ".LPL_LOOP:\n"
                    "PL_LD 72\n"
                    "PL_SEE 64\n"
                    "PL_LD 64\n"
                    "PL_SEE 68\n"
                    "PL_LD 68\n"
                    "PL_SEE 72\n"
                    "s_sub_u32 %[n], %[n], 1\n"
                    "s_cbranch_scc0 .LPL_LOOP\n"
                    ".LPL_END:\n"
                    "s_mov_b64 exec, %[sv]\n"
                    "s_waitcnt vmcnt(0)\n"
                    : [m] "=&s"(m_), [sv] "=&s"(sv_), [n] "=&s"(n_), [seen] "+s"(seen)
                    : [fa] "v"(fl + 4 * ws), [ta] "v"(tmo), [ia] "v"(lds0 + LDS_INFO + (4 * ws + lane) * 4), [call] "s"(call), [full] "s"(full)
                    : "memory", "vcc", "scc", "v64", "v65", "v66", "v68", "v69", "v70", "v72", "v73", "v74");
                static_assert(INFO_UP == 0x100u, "constant of the poll");
                if (seen == full) break;
            }
            const bool in = ws + (lane >> 2) <= lim;
            unsigned long long v = 0;
            if (in) v = __hip_atomic_load(fl + 4 * ws, __ATOMIC_RELAXED, __HIP_MEMORY_SCOPE_SYSTEM);
            const unsigned tm = __hip_atomic_load(tmo, __ATOMIC_RELAXED, __HIP_MEMORY_SCOPE_AGENT);
            if (__builtin_amdgcn_readfirstlane(tm) == call || __builtin_amdgcn_s_memrealtime() - t0 > DEADLINE_TICKS) {
                if (lane == 0) __hip_atomic_store(tmo, call, __ATOMIC_RELAXED, __HIP_MEMORY_SCOPE_AGENT);
                ctrl_set(ctrl + C_BAD * 4, 1u);
                ctrl_set(lds0 + LDS_INFO + lane * 4, INFO_SKIP);
                ctrl_set(lds0 + LDS_INFO + 256 + lane * 4, INFO_SKIP);
                break;
            }
            const bool up = (unsigned)(v >> 32) >= call;
            const unsigned long long m = __builtin_amdgcn_ballot_w64(up);
            if (m & ~seen) {
                if (up) ctrl_set(lds0 + LDS_INFO + (4 * ws + lane) * 4, (unsigned)v | INFO_UP);
                seen |= m;
                while ((seen & 15) == 15) { seen >>= 4; ++ws; }
            } else {
                __builtin_amdgcn_s_sleep(1);
            }
        }
    } else if (wave < 14) {
        const int ld = wave - 12, nld = 2;
        const int wr = lane >> 4;
        const unsigned wo0 = (unsigned)(wr * NLOC + n0 + (((lane & 15) ^ (2 * wr)) << 4));
        const unsigned wo1 = (unsigned)((wr + 4) * NLOC + n0 + (((lane & 15) ^ (2 * (wr + 4))) << 4)) - 1024u;
        const unsigned wo2 = wo0 + 8 * NLOC - 2048u;
        const unsigned wo3 = wo1 + 8 * NLOC - 2048u;
        __builtin_amdgcn_s_setprio(2);
        const unsigned long long watch_from = tstart + (ord0 * PERIOD / 100 - 150);
        for (int i = 0; i < ld; ++i) __builtin_amdgcn_s_sleep(16);
        for (int s = ld; s < NSTEP; s += nld) {
            const int sl = s % 3;
            for (;;) {
                const unsigned v = ctrl_get(ctrl + (lane & 31) * 4);
                const unsigned ok = (unsigned)__builtin_amdgcn_ballot_w64((int)v > s - 2);
                if ((ok & (0xFFu << C_DONE)) == (0xFFu << C_DONE) && __builtin_amdgcn_readlane(v, C_HOLD) == 0) break;
                __builtin_amdgcn_s_sleep(1);
            }
            const unsigned char* g = p.w + (size_t)s * (BK * NLOC);
            const unsigned l = lds0 + LDS_W + sl * W_STAGE;
            if (!local || __builtin_amdgcn_s_memrealtime() < watch_from) {
#pragma unroll
                for (int i = 0; i < 8; ++i) dma4<DMA_NOP>(g + i * 16 * NLOC, wo0, wo1, wo2, wo3, l + i * 4096);
            } else {
                unsigned hold = 0;
#pragma unroll
                for (int i = 0; i < 8; ++i) {
                    while (__builtin_amdgcn_readfirstlane(hold) != 0u) { __builtin_amdgcn_s_sleep(1); hold = ctrl_get(ctrl + C_HOLD * 4); }
                    hold = *(volatile LAS unsigned*)(size_t)(ctrl + C_HOLD * 4);
                    dma4<DMA_NOP_L>(g + i * 16 * NLOC, wo0, wo1, wo2, wo3, l + i * 4096);
                }
            }
            asm volatile("s_waitcnt vmcnt(0)" ::: "memory");
            ctrl_set1(ctrl + (C_WFULL + sl) * 4, (unsigned)(s + 1));
        }
    } else if (sender) {
        __builtin_amdgcn_s_setprio(3);
        const int lst_off = LDS_LIST + (15 - wave) * 256;
        const unsigned long long wsl = peer_of_lane();
        const unsigned rot = (unsigned)ord % (NR - 1);
        unsigned long long r1[NR], r2[NR], r4[NR];
#pragma unroll
        for (int j = 1; j < NR; ++j) { r1[j] = (rot & 1) ? peer[1 + j % (NR - 1)] : peer[j]; asm volatile("" : "+s"(r1[j])); }
#pragma unroll
        for (int j = 1; j < NR; ++j) { r2[j] = (rot & 2) ? r1[1 + (j + 1) % (NR - 1)] : r1[j]; asm volatile("" : "+s"(r2[j])); }
#pragma unroll
        for (int j = 1; j < NR; ++j) r4[j] = (rot & 4) ? r2[1 + (j + 3) % (NR - 1)] : r2[j];
        const unsigned call = __builtin_amdgcn_readfirstlane(cnt) + 1u;
        const unsigned par = call & 1u;
        const unsigned o = lane * 16;
        asm volatile("" : "+v"(x0), "+v"(x1), "+v"(x2), "+v"(x3));
        unsigned* tok = (unsigned*)(myws + WS_TOK);
        unsigned long long* late = (unsigned long long*)(myws + WS_LATE);
        const int xl = (int)(__builtin_amdgcn_s_memrealtime() - tstart) - X_NOM;
        const unsigned xlate = xl > 0 ? xl : 0;
        if (ord < 8 && xlate > 15 && lane == 0)
            __hip_atomic_fetch_max(late, ((unsigned long long)call << 32) | xlate, __ATOMIC_RELAXED, __HIP_MEMORY_SCOPE_AGENT);
        LAS u32x4* stash = (LAS u32x4*)(smem + LDS_STASH + (15 - wave) * PIECE + lane * 16);
        int s_piece = 0, s_lo = 0, s_nraw = 0;
        while (ord < ord_end) {
            bool raw = true;
            int lo = 0, nraw = ord < RAW_FIRST ? ord : RAW_FIRST;
            u32x4 d0 = x0, d1 = x1, d2 = x2, lst = x3;
            if (ahead == 3) {
                d0 = stash[0]; d1 = stash[64]; d2 = stash[128]; lst = stash[192];
                piece = s_piece; lo = s_lo; nraw = s_nraw; raw = lo < 0;
                ahead = 0;
            } else if (ord >= RAW_FIRST) {
                if (lane < 33) ctrl_set(lds0 + lst_off + lane * 4, 0u);
                unsigned sum = 0;
#pragma unroll
                for (int d = 0; d < 4; ++d) sum = __builtin_amdgcn_sad_u8(x0[d] & 0x7F7F7F7Fu, 0u, sum);
                sum += __builtin_amdgcn_update_dpp(0u, sum, 0x111, 0xf, 0xf, false);
                sum += __builtin_amdgcn_update_dpp(0u, sum, 0x112, 0xf, 0xf, false);
                sum += __builtin_amdgcn_update_dpp(0u, sum, 0x114, 0xf, 0xf, false);
                sum += __builtin_amdgcn_update_dpp(0u, sum, 0x118, 0xf, 0xf, false);
                const int tot = __builtin_amdgcn_readlane(sum, 15) + __builtin_amdgcn_readlane(sum, 31) +
                                __builtin_amdgcn_readlane(sum, 47) + __builtin_amdgcn_readlane(sum, 63);
                lo = (tot - 20582) >> 10;
                lo = lo < 0 ? 0 : (lo > 96 ? 96 : lo);
                const unsigned lo4 = (unsigned)lo * 0x01010101u;
                unsigned m[4];
                u32x4 c0, c1, c2;
#pragma unroll
                for (int i = 0; i < 4; ++i) {
                    const u32x3 c = pack((u32x4){x0[i], x1[i], x2[i], x3[i]}, lo4, m[i]);
                    c0[i] = c[0]; c1[i] = c[1]; c2[i] = c[2];
                }
                unsigned n = 0;
#pragma unroll
                for (int v = 0; v < 4; ++v) n += __builtin_popcount((m[v] | (m[v] >> 1)) & 0x55555555u);
                if (n) {
                    unsigned slot;
                    asm volatile("ds_add_rtn_u32 %0, %1, %2\n\ts_waitcnt lgkmcnt(0)" : "=&v"(slot) : "v"(lds0 + lst_off + 128), "v"(n) : "memory");
#pragma unroll
                    for (int v = 0; v < 4; ++v) {
                        unsigned mv = m[v];
                        while (mv != 0 && slot < LIST_CAP) {
                            const unsigned bit = (unsigned)__builtin_ctz(mv) & ~1u;
                            const unsigned dl = (mv >> bit) & 3u;
                            mv &= ~(3u << bit);
                            const unsigned pos = ((bit >> 1) & 3) * 1024 + lane * 16 + v * 4 + (bit >> 3);
                            *(LAS unsigned short*)(smem + lst_off + slot * 2) = (unsigned short)((pos << 2) | dl);
                            ++slot;
                        }
                    }
                }
                asm volatile("" ::: "memory");
                lst = *(volatile LAS u32x4*)(smem + lst_off + (lane & 7) * 16);
                const int npatch = __builtin_amdgcn_readfirstlane(ctrl_get(lds0 + lst_off + 128));
                raw = npatch > LIST_CAP;
                if (npatch > 2 * LIST_CAP) nraw = ord;
                if (!raw) { d0 = c0; d1 = c1; d2 = c2; } else lst = x3;
            }
            if (ahead == 1) {
                stash[0] = d0; stash[64] = d1; stash[128] = d2; stash[192] = lst;
                s_piece = piece; s_lo = raw ? -1 : lo; s_nraw = nraw;
                ahead = 2;
                ord -= ord_step;
                load_x();
                continue;
            }
            const int step = piece >> 4, grp = piece & 15;
            const size_t off = par * LAND_BYTES + (size_t)piece * PIECE;
            GAS unsigned long long* fp = (GAS unsigned long long*)(wsl + SRC_FLAG) + ((par * 4 + (grp >> 2)) * NSTEP + step) * 4 + (grp & 3);
            char* lp = (char*)(wsl + off + 3072 + (lane & 7) * 16);
            asm volatile("" : "+v"(d0), "+v"(d1), "+v"(d2), "+v"(lst), "+v"(lp));
            __builtin_amdgcn_s_setprio(3);
            unsigned long long due = tstart + (unsigned long long)(OFFSET + (ord * PERIOD + nraw * 160) / 100);
            if (ord >= SEND_WIN) {
                const unsigned long long due0 = due;
                while (__builtin_amdgcn_s_memrealtime() + 400 < due) __builtin_amdgcn_s_sleep(32);
                for (;;) {
                    while (__builtin_amdgcn_s_memrealtime() + (local ? HOLD_LEAD + 130 : HOLD_LEAD + 60) < due) __builtin_amdgcn_s_sleep(2);
                    const unsigned tk = __hip_atomic_load(tok + (ord - SEND_WIN - (local ? 24 : 0)) * 16, __ATOMIC_RELAXED, __HIP_MEMORY_SCOPE_AGENT);
                    const unsigned long long lt = __hip_atomic_load(late, __ATOMIC_RELAXED, __HIP_MEMORY_SCOPE_AGENT);
                    if (__builtin_amdgcn_readfirstlane((unsigned)(lt >> 32)) == call) due = due0 + __builtin_amdgcn_readfirstlane((unsigned)lt);
                    const unsigned long long now = __builtin_amdgcn_s_memrealtime();
                    if (now > due0 + 20000) break;
                    if (__builtin_amdgcn_readfirstlane(tk) >= call) {
                        if (now + (local ? HOLD_LEAD + 130 : HOLD_LEAD + 60) >= due) break;
                    } else if (now > due && lane == 0) {
                        __hip_atomic_fetch_max(late, ((unsigned long long)call << 32) | (unsigned)(now - due0), __ATOMIC_RELAXED, __HIP_MEMORY_SCOPE_AGENT);
                    }
                }
            } else {
                due += xlate;
                while (__builtin_amdgcn_s_memrealtime() + HOLD_LEAD + 10 < due) __builtin_amdgcn_s_sleep(2);
            }
            while (__builtin_amdgcn_s_memrealtime() + HOLD_LEAD < due) __builtin_amdgcn_s_sleep(1);
            if (lane == 0 && ord >= EARLY) asm volatile("ds_add_u32 %0, %1" :: "v"(ctrl + C_HOLD * 4), "v"(1u) : "memory");
            while (__builtin_amdgcn_s_memrealtime() < due) __builtin_amdgcn_s_sleep(1);
            const unsigned vo = (unsigned)off + o;
            st7<0>(r4, vo, d0);
            st7<1024>(r4, vo, d1);
            st7<2048>(r4, vo, d2);
            if (raw) st7<3072>(r4, vo, lst);
            if (!raw && lane >= 8) st_ptr(lp, lst);
            if (lane == 0) asm volatile("ds_sub_u32 %0, %1" :: "v"(ctrl + C_HOLD * 4), "v"(1u) : "memory");
            asm volatile("s_waitcnt vmcnt(0)" ::: "memory");
            if (lane >= 8 && (lane & 7) == 0)
                __hip_atomic_store(fp, ((unsigned long long)call << 32) | (raw ? INFO_RAW : (unsigned)lo),
                                   __ATOMIC_RELAXED, __HIP_MEMORY_SCOPE_SYSTEM);
            if (lane == 0) __hip_atomic_store(tok + ord * 16, call, __ATOMIC_RELAXED, __HIP_MEMORY_SCOPE_AGENT);
            ord += ord_step;
            if (ord >= ord_end) break;
            if (ahead == 2) { ahead = 3; continue; }
            if (!local) __builtin_amdgcn_s_setprio(0);
            load_x();
        }
        if (lane == 0) __hip_atomic_store(cntp, call, __ATOMIC_RELAXED, __HIP_MEMORY_SCOPE_AGENT);
    }
}

}

extern "C" void kernel_launch(int rank, int nranks, ncclComm_t comm,
                              hipStream_t stream,
                              void* const* d_in, const int* in_sizes, int n_in,
                              void* d_out, int out_size,
                              const int* in_dtypes, int out_dtype,
                              void* const* d_out_all,
                              void* const* d_ws_all, size_t ws_bytes) {
    (void)comm; (void)in_dtypes; (void)out_dtype; (void)d_out_all;
    if (nranks != NR || (unsigned)rank >= (unsigned)NR || n_in != 4 || ws_bytes < WS_NEED ||
        in_sizes[0] != MLOC * KDIM || in_sizes[1] != KDIM * NLOC || out_size != NR * MLOC * NLOC) {
        fprintf(stderr, "ag_gemm: unexpected problem shape\n");
        return;
    }
    HIP_CHECK(hipFuncSetAttribute((const void*)ag_gemm, hipFuncAttributeMaxDynamicSharedMemorySize, LDS_BYTES));
    Params p;
    p.x = (const unsigned char*)d_in[0];
    p.w = (const unsigned char*)d_in[1];
    p.sx = (const float*)d_in[2];
    p.sw = (const float*)d_in[3];
    p.out = (float*)d_out;
    for (int j = 0; j < NR; ++j) p.ws[j] = (char*)d_ws_all[(rank + j) % NR] + (j ? rank * SRC_BYTES : 0);
    p.rank = rank;
    p.pad = 0;
    hipLaunchKernelGGL(ag_gemm, dim3(GRID), dim3(NTHREADS), LDS_BYTES, stream, p);
    HIP_CHECK(hipGetLastError());
}
```
